```python
import numpy as np

kernel_name = "gpu_gemm_a2a_m4096_k4096_n4096_f32_relu"


ABSMAX_RTOL = 0.001

COMPUTE_DTYPE = "float32"
EPILOGUE = "relu"
IN_QUANT = None
OUT_QUANT = None
IN_DTYPES = {"x": "float32", "w_mat": "float32"}
WORLD_SIZE = 8

def setup_inputs():

    x = np.random.randn(4096, 4096).astype(np.float32)
    w_mat = (np.random.randn(4096, 4096) / np.sqrt(4096)).astype(np.float32)
    return {"x": x, "w_mat": w_mat}

def _epilogue(y):
    return np.maximum(y, 0.0).astype(np.float32)

def reference(x, w_mat):
    y = (x @ w_mat).astype(np.float32)
    return _epilogue(y)
```

```cpp
#include <hip/hip_runtime.h>
#include <rccl/rccl.h>
#include <cstdio>

#define HIP_CHECK(expr) do { hipError_t _e = (expr); if (_e != hipSuccess) { \
    fprintf(stderr, "HIP error %s at %s:%d\n", hipGetErrorString(_e), __FILE__, __LINE__); \
    return; } } while (0)

typedef float f32x4 __attribute__((ext_vector_type(4)));
typedef _Float16 f16x2 __attribute__((ext_vector_type(2)));
typedef unsigned u32x4 __attribute__((ext_vector_type(4)));
typedef unsigned long long u64;
#define LDS_AS __attribute__((address_space(3)))

constexpr int NRANK = 8;
constexpr int M_SH = 512, K_DIM = 4096, N_DIM = 4096, N_SH = 512;
constexpr int BM = 256, BN = 64;
constexpr int NWG = 256;
#define EARLY 4
#define SWR 8
#define KC 20
constexpr int KSV = 64 - 3 * KC;
constexpr int NST = 5;
constexpr u64 DEADLINE_TICKS = 50000000ull;
constexpr int WS_XFLAG = 4096;
constexpr int WS_SEQ = 49152;
constexpr int WS_DBG = 65536;
constexpr int WS_X16 = 1 << 20;
constexpr int WS_XCH = 10 << 20;
constexpr int MSG_BYTES = 4224;
constexpr int WS_MSG = 18 << 20;
constexpr int MSG_TILE = 16 * MSG_BYTES;
constexpr size_t WS_NEED = (size_t)WS_MSG + 2ull * 8 * 16 * MSG_TILE;
constexpr int RED_BYTES = 3 * 4096;
constexpr int NRING = 6;
constexpr int LDS_OWN = NRING * RED_BYTES;
constexpr int LDS_STAGE = LDS_OWN + 16 * 4096;
constexpr int LDS_CNT = LDS_STAGE + 4 * 5120;
constexpr int LDS_BYTES = LDS_CNT + 256;
constexpr unsigned OOB = 0x7ffffff0u;

struct Params {
  const float* x; const float* w;
  float* out;
  char* myws;
  char* ws[NRANK];
  int rank;
};

__device__ __forceinline__ unsigned pk(float a, float b) {
  f16x2 r; r[0] = (_Float16)a; r[1] = (_Float16)b; return __builtin_bit_cast(unsigned, r);
}
__device__ __forceinline__ u32x4 make_srd(const void* p, unsigned bytes) {
  const u64 a = (u64)p;
  u32x4 r;
  r[0] = __builtin_amdgcn_readfirstlane((unsigned)a);
  r[1] = __builtin_amdgcn_readfirstlane((unsigned)(a >> 32)) & 0xffffu;
  r[2] = bytes; r[3] = 0x00020000u;
  return r;
}
__device__ __forceinline__ unsigned wave_scan(unsigned v) {
  v += __builtin_amdgcn_update_dpp(0u, v, 0x111, 0xf, 0xf, false);
  v += __builtin_amdgcn_update_dpp(0u, v, 0x112, 0xf, 0xf, false);
  v += __builtin_amdgcn_update_dpp(0u, v, 0x114, 0xf, 0xf, false);
  v += __builtin_amdgcn_update_dpp(0u, v, 0x118, 0xf, 0xf, false);
  v += __builtin_amdgcn_update_dpp(0u, v, 0x142, 0xa, 0xf, false);
  v += __builtin_amdgcn_update_dpp(0u, v, 0x143, 0xc, 0xf, false);
  return v;
}
#define STAMP(i)
#define STAMP_FLUSH()
#define STAMP_DECL
#define SB() __builtin_amdgcn_sched_barrier(0)
#define TIMED_OUT() (__builtin_amdgcn_s_memrealtime() - t_start > DEADLINE_TICKS)

__global__ void __launch_bounds__(256, 1) gemm_a2a(Params P) {
  extern __shared__ __attribute__((aligned(16))) char smem[];
  const int tid = threadIdx.x, lane = tid & 63, wv = __builtin_amdgcn_readfirstlane(tid >> 6);
  const int bid = blockIdx.x;
  const int idx = bid >> 3;
  const int kap = bid & 1, rho = idx & 1, dd = idx >> 2;
  const int strip = (bid & 6) + ((idx >> 1) & 1);
  const int dest = (P.rank + dd) & 7, src = (P.rank - dd) & 7;
  const int tile = rho * 8 + strip, pid = dd * 16 + tile;
  char* myws = P.myws;
  const int c = lane & 15, g = lane >> 4;
  STAMP_DECL
  STAMP(0);
  const u64 t_start = __builtin_amdgcn_s_memrealtime();
  volatile LDS_AS unsigned* cnts = (volatile LDS_AS unsigned*)(smem + LDS_CNT);
  const char* seqp = myws + WS_SEQ + 64 * bid;
  unsigned call;
  if (tid < 32) ((unsigned*)(smem + LDS_CNT))[tid] = 0u;

  const u32x4 rw = make_srd(P.w, K_DIM * N_DIM * 4);
  u32x4 rxs;
#define BLOAD(dst, vo, rs, so, imm, aux) asm volatile("buffer_load_dwordx4 %0, %1, %2, %3 offen offset:%4" aux : "=v"(dst) : "v"(vo), "s"(rs), "s"(so), "n"(imm) : "memory")
#define BSTORE(v, vo, rs, so, imm, aux) asm volatile("buffer_store_dwordx4 %0, %1, %2, %3 offen offset:%4" aux "\n\ts_nop 1" :: "v"(v), "v"(vo), "s"(rs), "s"(so), "n"(imm) : "memory")
#define WAIT1(n, a) asm volatile("s_waitcnt vmcnt(%1)" : "+v"(a) : "n"(n) : "memory")
#define WAIT2(n, a, b) asm volatile("s_waitcnt vmcnt(%2)" : "+v"(a), "+v"(b) : "n"(n) : "memory")
  const unsigned wvo = (unsigned)(8 * g * N_DIM + dest * N_SH + strip * BN + 4 * c) * 4u;
  const unsigned wso = (unsigned)(kap * 2048 + wv * KC * 32) * (N_DIM * 4u);
  const unsigned xvo = lane * 16;
  bool bad = false;
  unsigned* const xflag = (unsigned*)(myws + WS_XFLAG + 8192 * (rho * 2 + kap) + 128 * lane);
#define WLOAD(i) BLOAD(wl[i], wvo, rw, wso + (unsigned)(((i) >> 3) * 32 + ((i) & 7)) * (N_DIM * 4u), 0, "")
#define XLOAD(s, q) BLOAD(win[s], xvo, rxs, (unsigned)(((q) >> 2) * 4096), ((q) & 3) * 1024, " sc1")
#define WCVT(i, jj) do { _Pragma("unroll") for (int t = 0; t < 4; ++t) { bb[t][jj] = pk(wl[i][t], wl[(i) + 1][t]); asm volatile("" : "+v"(bb[t][jj])); } SB(); } while (0)
#define WKEEP_(C, ks) do { _Pragma("unroll") for (int t = 0; t < 4; ++t) { u32x4 b_ = {bb[t][0], bb[t][1], bb[t][2], bb[t][3]}; \
    asm volatile("" : C(b_)); breg[ks][t] = b_; } SB(); } while (0)
#define WKEEPA(ks) WKEEP_("+a", ks)
#define WKEEPV(ks) WKEEP_("+v", ks)
#define GETCALL() do { asm volatile("s_load_dword %0, %1, 0x0\n\ts_waitcnt lgkmcnt(0)" : "=s"(call) : "s"(seqp) : "memory"); call += 1u; \
    rxs = make_srd(myws + WS_X16 + (size_t)(call & 1) * (M_SH * K_DIM * 2) + (rho * 2 + kap) * 1048576 + wv * (16 * KC * 1024), \
                   (wv < 3 ? 16 * KC : 16 * KSV) * 1024u);                                                           \
    __builtin_amdgcn_s_barrier();                                                            \
    asm volatile("s_nop 4" ::: "memory"); } while (0)
#define XPOLL() do { while (!__all(__hip_atomic_load(xflag, __ATOMIC_RELAXED, __HIP_MEMORY_SCOPE_AGENT) >= call)) {   \
      if (TIMED_OUT()) { bad = true; break; }                                                                        \
      __builtin_amdgcn_s_sleep(4); } } while (0)
#define MF(i, CIN) "v_mfma_f32_16x16x32_f16 %[c" #i "], %[a], %[b" #i "], " CIN(i) "\n\t"
#define CIN_ACC(i) "%[c" #i "]"
#define CIN_ZERO(i) "0"
#define COUT(CC, A) [c0] CC(A[0]), [c1] CC(A[1]), [c2] CC(A[2]), [c3] CC(A[3])
#define BC_A "a"
#define BC_V "v"
#define BIN(K, ks) [b0] BC_##K(breg[ks][0]), [b1] BC_##K(breg[ks][1]), [b2] BC_##K(breg[ks][2]), [b3] BC_##K(breg[ks][3])
#define STEP_(K, CIN, CC, A, n_, s, sp, sn, so_, imm_, ks) do { asm volatile(MF(0, CIN)                               \
    "buffer_load_dwordx4 %[p], %[vo], %[rs], %[so] offen offset:%[imm] sc1\n\t" MF(1, CIN) MF(2, CIN)                \
    "s_waitcnt vmcnt(%[n])\n\t" MF(3, CIN)                                                                           \
    : COUT(CC, A), [p] "+v"(win[sp]), [nx] "+v"(win[sn])                                                             \
    : [a] "v"(win[s]), BIN(K, ks), [vo] "v"(xvo), [rs] "s"(rxs), [so] "s"(so_), [imm] "n"(imm_), [n] "n"(n_) : "memory"); SB(); } while (0)
#define MFMA_FENCE(A) asm volatile("s_nop 7\n\ts_nop 1" : "+v"(A[0]), "+v"(A[1]), "+v"(A[2]), "+v"(A[3]))

  if (wv < 3) {
    f32x4 wl[8 * KC];
    u32x4 win[KC];
    u32x4 breg[KC][4];
    f32x4 eacc[EARLY][4];
    unsigned xc = 0;
#define P1_INIT() GETCALL()
#define P1_CLOAD() do { asm volatile("global_load_dword %0, %1, off sc1" : "=v"(xc) : "v"(xflag) : "memory"); SB(); } while (0)
#define P1_CHECK() do { asm volatile("" : "+v"(xc)); if (!__all(xc >= call)) XPOLL(); asm volatile("" ::: "memory"); SB(); STAMP(1); } while (0)
#define EMFMA_(K, CIN, CC, n_, s, e, ks) do { asm volatile("s_waitcnt vmcnt(%[n])\n\ts_nop 0\n\t" MF(0, CIN) MF(1, CIN) MF(2, CIN) MF(3, CIN) \
    : COUT(CC, eacc[e]), [a] "+v"(win[s]) : BIN(K, ks), [n] "n"(n_) : "memory"); SB(); } while (0)
#define EMFMA0(K, n_, s, e, ks) EMFMA_(K, CIN_ZERO, "=&v", n_, s, e, ks)
#define EMFMA(K, n_, s, e, ks) EMFMA_(K, CIN_ACC, "+v", n_, s, e, ks)
#define ESTEP(K, n_, s, sp, sn, q, e, ks) STEP_(K, CIN_ACC, "+v", eacc[e], n_, s, sp, sn, (unsigned)(((q) >> 2) * 4096), ((q) & 3) * 1024, ks)
    asm volatile("s_nop 4" ::: "memory");
    WLOAD(0);
    WLOAD(1);
    WLOAD(2);
    WLOAD(3);
    WLOAD(4);
    WLOAD(5);
    WLOAD(6);
    WLOAD(7);
    SB();
    P1_INIT();
    { unsigned bb[4][4];
    WAIT2(6, wl[0], wl[1]);
    WCVT(0, 0);
    WLOAD(8);
    WLOAD(9);
    SB();
    WAIT2(6, wl[2], wl[3]);
    WCVT(2, 1);
    WLOAD(10);
    WLOAD(11);
    SB();
    WAIT2(6, wl[4], wl[5]);
    WCVT(4, 2);
    WLOAD(12);
    WLOAD(13);
    SB();
    WAIT2(6, wl[6], wl[7]);
    WCVT(6, 3);
    WLOAD(14);
    WLOAD(15);
    SB();
    WKEEPA(0); }
    { unsigned bb[4][4];
    WAIT2(6, wl[8], wl[9]);
    WCVT(8, 0);
    WLOAD(16);
    WLOAD(17);
    SB();
    WAIT2(6, wl[10], wl[11]);
    WCVT(10, 1);
    WLOAD(18);
    WLOAD(19);
    SB();
    WAIT2(6, wl[12], wl[13]);
    WCVT(12, 2);
    WLOAD(20);
    WLOAD(21);
    SB();
    WAIT2(6, wl[14], wl[15]);
    WCVT(14, 3);
    WLOAD(22);
    WLOAD(23);
    SB();
    WKEEPA(1); }
    { unsigned bb[4][4];
    WAIT2(6, wl[16], wl[17]);
    WCVT(16, 0);
    WLOAD(24);
    WLOAD(25);
    SB();
    WAIT2(6, wl[18], wl[19]);
    WCVT(18, 1);
    WLOAD(26);
    WLOAD(27);
    SB();
    WAIT2(6, wl[20], wl[21]);
    WCVT(20, 2);
    WLOAD(28);
    WLOAD(29);
    SB();
    WAIT2(6, wl[22], wl[23]);
    WCVT(22, 3);
    WLOAD(30);
    WLOAD(31);
    SB();
    WKEEPA(2); }
    { unsigned bb[4][4];
    WAIT2(6, wl[24], wl[25]);
    WCVT(24, 0);
    WLOAD(32);
    WLOAD(33);
    SB();
    WAIT2(6, wl[26], wl[27]);
    WCVT(26, 1);
    WLOAD(34);
    WLOAD(35);
    SB();
    WAIT2(6, wl[28], wl[29]);
    WCVT(28, 2);
    WLOAD(36);
    WLOAD(37);
    SB();
    WAIT2(6, wl[30], wl[31]);
    WCVT(30, 3);
    WLOAD(38);
    WLOAD(39);
    SB();
    WKEEPA(3); }
    { unsigned bb[4][4];
    WAIT2(6, wl[32], wl[33]);
    WCVT(32, 0);
    WLOAD(40);
    WLOAD(41);
    SB();
    WAIT2(6, wl[34], wl[35]);
    WCVT(34, 1);
    WLOAD(42);
    WLOAD(43);
    SB();
    WAIT2(6, wl[36], wl[37]);
    WCVT(36, 2);
    WLOAD(44);
    WLOAD(45);
    SB();
    WAIT2(6, wl[38], wl[39]);
    WCVT(38, 3);
    WLOAD(46);
    WLOAD(47);
    SB();
    WKEEPA(4); }
    { unsigned bb[4][4];
    WAIT2(6, wl[40], wl[41]);
    WCVT(40, 0);
    WLOAD(48);
    WLOAD(49);
    SB();
    WAIT2(6, wl[42], wl[43]);
    WCVT(42, 1);
    WLOAD(50);
    WLOAD(51);
    SB();
    WAIT2(6, wl[44], wl[45]);
    WCVT(44, 2);
    WLOAD(52);
    WLOAD(53);
    SB();
    P1_CLOAD();
    WAIT2(7, wl[46], wl[47]);
    WCVT(46, 3);
    WLOAD(54);
    WLOAD(55);
    SB();
    WKEEPA(5); }
    { unsigned bb[4][4];
    WAIT2(7, wl[48], wl[49]);
    WCVT(48, 0);
    WLOAD(56);
    WLOAD(57);
    SB();
    WAIT2(7, wl[50], wl[51]);
    WCVT(50, 1);
    WLOAD(58);
    WLOAD(59);
    SB();
    WAIT2(7, wl[52], wl[53]);
    WCVT(52, 2);
    WLOAD(60);
    WLOAD(61);
    SB();
    WAIT2(6, wl[54], wl[55]);
    WCVT(54, 3);
    WLOAD(62);
    WLOAD(63);
    SB();
    WKEEPA(6); }
    P1_CHECK();
    XLOAD(0, 0);
    XLOAD(1, 1);
    { unsigned bb[4][4];
    WAIT2(8, wl[56], wl[57]);
    WCVT(56, 0);
    WLOAD(64);
    WLOAD(65);
    SB();
    XLOAD(2, 2);
    XLOAD(3, 3);
    WAIT2(10, wl[58], wl[59]);
    WCVT(58, 1);
    WLOAD(66);
    WLOAD(67);
    SB();
    XLOAD(4, 4);
    XLOAD(5, 5);
    WAIT2(12, wl[60], wl[61]);
    WCVT(60, 2);
    WLOAD(68);
    WLOAD(69);
    SB();
    XLOAD(6, 6);
    XLOAD(7, 7);
    WAIT2(14, wl[62], wl[63]);
    WCVT(62, 3);
    WLOAD(70);
    WLOAD(71);
    SB();
    WKEEPA(7); }
    EMFMA0(A, 15, 0, 0, 0);
    EMFMA0(A, 14, 1, 1, 0);
    XLOAD(8, 8);
    XLOAD(9, 9);
    { unsigned bb[4][4];
    WAIT2(14, wl[64], wl[65]);
    WCVT(64, 0);
    WLOAD(72);
    WLOAD(73);
    SB();
    EMFMA0(A, 15, 2, 2, 0);
    EMFMA0(A, 14, 3, 3, 0);
    XLOAD(10, 10);
    XLOAD(11, 11);
    WAIT2(14, wl[66], wl[67]);
    WCVT(66, 1);
    WLOAD(74);
    WLOAD(75);
    SB();
    EMFMA(A, 15, 4, 0, 1);
    EMFMA(A, 14, 5, 1, 1);
    XLOAD(12, 12);
    XLOAD(13, 13);
    WAIT2(14, wl[68], wl[69]);
    WCVT(68, 2);
    WLOAD(76);
    WLOAD(77);
    SB();
    EMFMA(A, 15, 6, 2, 1);
    EMFMA(A, 14, 7, 3, 1);
    XLOAD(14, 14);
    XLOAD(15, 15);
    WAIT2(14, wl[70], wl[71]);
    WCVT(70, 3);
    WLOAD(78);
    WLOAD(79);
    SB();
    WKEEPA(8); }
    EMFMA(A, 15, 8, 0, 2);
    EMFMA(A, 14, 9, 1, 2);
    XLOAD(16, 16);
    XLOAD(17, 17);
    { unsigned bb[4][4];
    WAIT2(14, wl[72], wl[73]);
    WCVT(72, 0);
    WLOAD(80);
    WLOAD(81);
    SB();
    EMFMA(A, 15, 10, 2, 2);
    EMFMA(A, 14, 11, 3, 2);
    XLOAD(18, 18);
    XLOAD(19, 19);
    WAIT2(14, wl[74], wl[75]);
    WCVT(74, 1);
    WLOAD(82);
    WLOAD(83);
    SB();
    EMFMA(A, 15, 12, 0, 3);
    EMFMA(A, 14, 13, 1, 3);
    XLOAD(0, 20);
    XLOAD(1, 21);
    WAIT2(14, wl[76], wl[77]);
    WCVT(76, 2);
    WLOAD(84);
    WLOAD(85);
    SB();
    EMFMA(A, 15, 14, 2, 3);
    EMFMA(A, 14, 15, 3, 3);
    XLOAD(2, 22);
    XLOAD(3, 23);
    WAIT2(14, wl[78], wl[79]);
    WCVT(78, 3);
    WLOAD(86);
    WLOAD(87);
    SB();
    WKEEPA(9); }
    EMFMA(A, 15, 16, 0, 4);
    EMFMA(A, 14, 17, 1, 4);
    XLOAD(4, 24);
    XLOAD(5, 25);
    { unsigned bb[4][4];
    WAIT2(14, wl[80], wl[81]);
    WCVT(80, 0);
    WLOAD(88);
    WLOAD(89);
    SB();
    EMFMA(A, 15, 18, 2, 4);
    EMFMA(A, 14, 19, 3, 4);
    XLOAD(6, 26);
    XLOAD(7, 27);
    WAIT2(14, wl[82], wl[83]);
    WCVT(82, 1);
    WLOAD(90);
    WLOAD(91);
    SB();
    EMFMA(A, 15, 0, 0, 5);
    EMFMA(A, 14, 1, 1, 5);
    XLOAD(8, 28);
    XLOAD(9, 29);
    WAIT2(14, wl[84], wl[85]);
    WCVT(84, 2);
    WLOAD(92);
    WLOAD(93);
    SB();
    EMFMA(A, 15, 2, 2, 5);
    EMFMA(A, 14, 3, 3, 5);
    XLOAD(10, 30);
    XLOAD(11, 31);
    WAIT2(14, wl[86], wl[87]);
    WCVT(86, 3);
    WLOAD(94);
    WLOAD(95);
    SB();
    WKEEPA(10); }
    EMFMA(A, 15, 4, 0, 6);
    EMFMA(A, 14, 5, 1, 6);
    XLOAD(12, 32);
    XLOAD(13, 33);
    { unsigned bb[4][4];
    WAIT2(14, wl[88], wl[89]);
    WCVT(88, 0);
    WLOAD(96);
    WLOAD(97);
    SB();
    EMFMA(A, 15, 6, 2, 6);
    EMFMA(A, 14, 7, 3, 6);
    XLOAD(14, 34);
    XLOAD(15, 35);
    WAIT2(14, wl[90], wl[91]);
    WCVT(90, 1);
    WLOAD(98);
    WLOAD(99);
    SB();
    EMFMA(A, 15, 8, 0, 7);
    EMFMA(A, 14, 9, 1, 7);
    XLOAD(16, 36);
    XLOAD(17, 37);
    WAIT2(14, wl[92], wl[93]);
    WCVT(92, 2);
    WLOAD(100);
    WLOAD(101);
    SB();
    EMFMA(A, 15, 10, 2, 7);
    EMFMA(A, 14, 11, 3, 7);
    XLOAD(18, 38);
    XLOAD(19, 39);
    WAIT2(14, wl[94], wl[95]);
    WCVT(94, 3);
    WLOAD(102);
    WLOAD(103);
    SB();
    WKEEPA(11); }
    EMFMA(A, 15, 12, 0, 8);
    EMFMA(A, 14, 13, 1, 8);
    XLOAD(0, 40);
    XLOAD(1, 41);
    { unsigned bb[4][4];
    WAIT2(14, wl[96], wl[97]);
    WCVT(96, 0);
    WLOAD(104);
    WLOAD(105);
    SB();
    EMFMA(A, 15, 14, 2, 8);
    EMFMA(A, 14, 15, 3, 8);
    XLOAD(2, 42);
    XLOAD(3, 43);
    WAIT2(14, wl[98], wl[99]);
    WCVT(98, 1);
    WLOAD(106);
    WLOAD(107);
    SB();
    EMFMA(A, 15, 16, 0, 9);
    EMFMA(A, 14, 17, 1, 9);
    XLOAD(4, 44);
    XLOAD(5, 45);
    WAIT2(14, wl[100], wl[101]);
    WCVT(100, 2);
    WLOAD(108);
    WLOAD(109);
    SB();
    EMFMA(A, 15, 18, 2, 9);
    EMFMA(A, 14, 19, 3, 9);
    XLOAD(6, 46);
    XLOAD(7, 47);
    WAIT2(14, wl[102], wl[103]);
    WCVT(102, 3);
    WLOAD(110);
    WLOAD(111);
    SB();
    WKEEPA(12); }
    EMFMA(A, 15, 0, 0, 10);
    EMFMA(A, 14, 1, 1, 10);
    XLOAD(8, 48);
    XLOAD(9, 49);
    { unsigned bb[4][4];
    WAIT2(14, wl[104], wl[105]);
    WCVT(104, 0);
    WLOAD(112);
    WLOAD(113);
    SB();
    EMFMA(A, 15, 2, 2, 10);
    EMFMA(A, 14, 3, 3, 10);
    XLOAD(10, 50);
    XLOAD(11, 51);
    WAIT2(14, wl[106], wl[107]);
    WCVT(106, 1);
    WLOAD(114);
    WLOAD(115);
    SB();
    EMFMA(A, 15, 4, 0, 11);
    EMFMA(A, 14, 5, 1, 11);
    XLOAD(12, 52);
    XLOAD(13, 53);
    WAIT2(14, wl[108], wl[109]);
    WCVT(108, 2);
    WLOAD(116);
    WLOAD(117);
    SB();
    EMFMA(A, 15, 6, 2, 11);
    EMFMA(A, 14, 7, 3, 11);
    XLOAD(14, 54);
    XLOAD(15, 55);
    WAIT2(14, wl[110], wl[111]);
    WCVT(110, 3);
    WLOAD(118);
    WLOAD(119);
    SB();
    WKEEPA(13); }
    EMFMA(A, 15, 8, 0, 12);
    EMFMA(A, 14, 9, 1, 12);
    XLOAD(16, 56);
    XLOAD(17, 57);
    { unsigned bb[4][4];
    WAIT2(14, wl[112], wl[113]);
    WCVT(112, 0);
    WLOAD(120);
    WLOAD(121);
    SB();
    EMFMA(A, 15, 10, 2, 12);
    EMFMA(A, 14, 11, 3, 12);
    XLOAD(18, 58);
    XLOAD(19, 59);
    WAIT2(14, wl[114], wl[115]);
    WCVT(114, 1);
    WLOAD(122);
    WLOAD(123);
    SB();
    EMFMA(A, 15, 12, 0, 13);
    EMFMA(A, 14, 13, 1, 13);
    XLOAD(0, 60);
    XLOAD(1, 61);
    WAIT2(14, wl[116], wl[117]);
    WCVT(116, 2);
    WLOAD(124);
    WLOAD(125);
    SB();
    EMFMA(A, 15, 14, 2, 13);
    EMFMA(A, 14, 15, 3, 13);
    XLOAD(2, 62);
    XLOAD(3, 63);
    WAIT2(14, wl[118], wl[119]);
    WCVT(118, 3);
    WLOAD(126);
    WLOAD(127);
    SB();
    WKEEPA(14); }
    EMFMA(A, 15, 16, 0, 14);
    EMFMA(A, 14, 17, 1, 14);
    XLOAD(4, 64);
    XLOAD(5, 65);
    { unsigned bb[4][4];
    WAIT2(14, wl[120], wl[121]);
    WCVT(120, 0);
    WLOAD(128);
    WLOAD(129);
    SB();
    EMFMA(A, 15, 18, 2, 14);
    EMFMA(A, 14, 19, 3, 14);
    XLOAD(6, 66);
    XLOAD(7, 67);
    WAIT2(14, wl[122], wl[123]);
    WCVT(122, 1);
    WLOAD(130);
    WLOAD(131);
    SB();
    XLOAD(8, 68);
    XLOAD(9, 69);
    WAIT2(14, wl[124], wl[125]);
    WCVT(124, 2);
    WLOAD(132);
    WLOAD(133);
    SB();
    XLOAD(10, 70);
    XLOAD(11, 71);
    WAIT2(14, wl[126], wl[127]);
    WCVT(126, 3);
    WLOAD(134);
    WLOAD(135);
    SB();
    WKEEPA(15); }
    EMFMA(A, 23, 0, 0, 15);
    EMFMA(A, 22, 1, 1, 15);
    XLOAD(12, 72);
    XLOAD(13, 73);
    { unsigned bb[4][4];
    WAIT2(14, wl[128], wl[129]);
    WCVT(128, 0);
    WLOAD(136);
    WLOAD(137);
    SB();
    EMFMA(A, 23, 2, 2, 15);
    EMFMA(A, 22, 3, 3, 15);
    XLOAD(14, 74);
    XLOAD(15, 75);
    WAIT2(14, wl[130], wl[131]);
    WCVT(130, 1);
    WLOAD(138);
    WLOAD(139);
    SB();
    XLOAD(16, 76);
    XLOAD(17, 77);
    WAIT2(14, wl[132], wl[133]);
    WCVT(132, 2);
    WLOAD(140);
    WLOAD(141);
    SB();
    XLOAD(18, 78);
    XLOAD(19, 79);
    WAIT2(14, wl[134], wl[135]);
    WCVT(134, 3);
    WLOAD(142);
    WLOAD(143);
    SB();
    WKEEPV(16); }
    EMFMA(V, 31, 4, 0, 16);
    EMFMA(V, 30, 5, 1, 16);
    XLOAD(0, 80);
    XLOAD(1, 81);
    { unsigned bb[4][4];
    WAIT2(14, wl[136], wl[137]);
    WCVT(136, 0);
    WLOAD(144);
    WLOAD(145);
    SB();
    EMFMA(V, 31, 6, 2, 16);
    EMFMA(V, 30, 7, 3, 16);
    XLOAD(2, 82);
    XLOAD(3, 83);
    WAIT2(14, wl[138], wl[139]);
    WCVT(138, 1);
    WLOAD(146);
    WLOAD(147);
    SB();
    XLOAD(4, 84);
    XLOAD(5, 85);
    WAIT2(14, wl[140], wl[141]);
    WCVT(140, 2);
    WLOAD(148);
    WLOAD(149);
    SB();
    XLOAD(6, 86);
    XLOAD(7, 87);
    WAIT2(14, wl[142], wl[143]);
    WCVT(142, 3);
    WLOAD(150);
    WLOAD(151);
    SB();
    WKEEPV(17); }
    EMFMA(V, 39, 8, 0, 17);
    EMFMA(V, 38, 9, 1, 17);
    XLOAD(8, 88);
    XLOAD(9, 89);
    { unsigned bb[4][4];
    WAIT2(14, wl[144], wl[145]);
    WCVT(144, 0);
    WLOAD(152);
    WLOAD(153);
    SB();
    EMFMA(V, 39, 10, 2, 17);
    EMFMA(V, 38, 11, 3, 17);
    XLOAD(10, 90);
    XLOAD(11, 91);
    WAIT2(14, wl[146], wl[147]);
    WCVT(146, 1);
    WLOAD(154);
    WLOAD(155);
    SB();
    WAIT2(12, wl[148], wl[149]);
    WCVT(148, 2);
    WLOAD(156);
    WLOAD(157);
    SB();
    WAIT2(10, wl[150], wl[151]);
    WCVT(150, 3);
    WLOAD(158);
    WLOAD(159);
    SB();
    WKEEPV(18); }
    EMFMA(V, 43, 12, 0, 18);
    EMFMA(V, 42, 13, 1, 18);
    XLOAD(12, 92);
    XLOAD(13, 93);
    { unsigned bb[4][4];
    WAIT2(10, wl[152], wl[153]);
    WCVT(152, 0);
    SB();
    EMFMA(V, 41, 14, 2, 18);
    EMFMA(V, 40, 15, 3, 18);
    XLOAD(14, 94);
    WAIT2(7, wl[154], wl[155]);
    WCVT(154, 1);
    SB();
    WAIT2(5, wl[156], wl[157]);
    WCVT(156, 2);
    SB();
    WAIT2(3, wl[158], wl[159]);
    WCVT(158, 3);
    SB();
    WKEEPV(19); }
    WAIT1(38, win[16]);
    asm volatile("s_nop 1" ::: "memory");
    ESTEP(V, 38, 16, 15, 17, 95, 0, 19);
    ESTEP(V, 36, 17, 16, 18, 96, 1, 19);
    ESTEP(V, 36, 18, 17, 19, 97, 2, 19);
    ESTEP(V, 34, 19, 18, 0, 98, 3, 19);
    STAMP(2);
    auto put_partial = [&](int o, const f32x4 (&acc)[4]) {
      f32x4* pw = (f32x4*)(smem + (o % NRING) * RED_BYTES + wv * 4096 + xvo);
#pragma unroll
      for (int t = 0; t < 4; ++t) pw[t * 64] = acc[t];
      asm volatile("" ::: "memory");
      if (lane == 0) asm volatile("ds_add_u32 %0, %1" :: "v"((unsigned)(size_t)(LDS_AS char*)(smem + LDS_CNT) + o * 4u), "v"(1u) : "memory");
      SB();
    };
    MFMA_FENCE(eacc[EARLY - 1]);
#pragma unroll
    for (int e = 0; e < EARLY; ++e) put_partial(e, eacc[e]);
    for (int o = EARLY; o < 16; ++o) {
      const unsigned qb = (unsigned)(o * KC + KC - 1) * 1024u;
      f32x4 acc[4];
      const unsigned consumed = cnts[16];
#pragma unroll
      for (int k = 0; k < KC; ++k) {
        if (k == 0) STEP_(A, CIN_ZERO, "=&v", acc, KC - 2, k, (k + KC - 1) % KC, (k + 1) % KC, qb + (k >> 2) * 4096, (k & 3) * 1024, k);
        else if (k < 16) STEP_(A, CIN_ACC, "+v", acc, KC - 2, k, (k + KC - 1) % KC, (k + 1) % KC, qb + (k >> 2) * 4096, (k & 3) * 1024, k);
        else STEP_(V, CIN_ACC, "+v", acc, KC - 2, k, (k + KC - 1) % KC, (k + 1) % KC, qb + (k >> 2) * 4096, (k & 3) * 1024, k);
      }
      MFMA_FENCE(acc);
      if ((int)__builtin_amdgcn_readfirstlane(consumed) < o - (NRING - 1)) {
        while ((int)cnts[16] < o - (NRING - 1)) { if (TIMED_OUT()) { bad = true; break; } }
      }
      asm volatile("" ::: "memory");
      put_partial(o, acc);
    }
    STAMP(4);
    asm volatile("s_waitcnt vmcnt(0)" ::: "memory");
#pragma unroll
    for (int i = 0; i < KC; ++i) asm volatile("" :: "v"(win[i]));
  } else {
    constexpr int SW = 2 * KSV;
    f32x4 wl[8 * KSV];
    u32x4 win[SW];
    u32x4 breg[KSV][4];
    {
      const int rc = bid >> 3, kb = bid & 7;
      const u32x4 rx = make_srd(P.x, M_SH * K_DIM * 4);
      const unsigned xcvo = (unsigned)((rc * 16 + c) * K_DIM + kb * 512 + g * 8) * 4u;
      f32x4 xl[32];
      asm volatile("s_nop 4" ::: "memory");
#pragma unroll
      for (int i = 0; i < 32; ++i) BLOAD(xl[i], xcvo, rx, 0u, (i >> 1) * 128 + (i & 1) * 16, "");
      GETCALL();
      const int grp = (rc >> 4) * 2 + (kb >> 2), oc = (rc & 15) ^ ((kb >> 2) ^ 1);
      const u32x4 rx16w = make_srd(myws + WS_X16 + (size_t)(call & 1) * (M_SH * K_DIM * 2) + grp * 1048576, 1048576u);
      asm volatile("s_nop 4" ::: "memory");
#pragma unroll
      for (int i = 0; i < 16; ++i) {
        const int gk = (kb & 3) * 16 + i;
        const int u = gk >= 3 * KC ? 3 : gk >= 2 * KC ? 2 : gk >= KC ? 1 : 0;
        const int ks = gk - u * KC;
        const int pos = u * 16 * KC + (u == 3 ? oc * KSV + ks : oc < EARLY ? ks * EARLY + oc : oc * KC + ks);
        WAIT2(30 - i, xl[2 * i], xl[2 * i + 1]);
        const f32x4 a_ = xl[2 * i], b_ = xl[2 * i + 1];
        u32x4 o_ = {pk(a_[0], a_[1]), pk(a_[2], a_[3]), pk(b_[0], b_[1]), pk(b_[2], b_[3])};
        BSTORE(o_, xvo, rx16w, (unsigned)pos * 1024u, 0, " sc1");
      }
      SB();
#pragma unroll
      for (int i = 0; i < SWR; ++i) WLOAD(i);
      asm volatile("s_waitcnt vmcnt(%0)" :: "n"(SWR) : "memory");
      if (lane == 0) {
        unsigned* f_ = (unsigned*)(myws + WS_XFLAG + 8192 * grp + 128 * ((rc & 15) * 4 + (kb & 3)));
        asm volatile("global_store_dword %0, %1, off sc1" :: "v"(f_), "v"(call) : "memory");
      }
      SB();
      STAMP(6);
    }
#pragma unroll
    for (int ks = 0; ks < KSV; ++ks) {
      unsigned bb[4][4];
#pragma unroll
      for (int jj = 0; jj < 4; ++jj) {
        const int i = ks * 8 + 2 * jj;
        WAIT2(i + SWR <= 8 * KSV ? SWR - 2 : 8 * KSV - 2 - i, wl[i], wl[i + 1]);
        WCVT(i, jj);
        if (i + SWR < 8 * KSV) { WLOAD(i + SWR); WLOAD(i + SWR + 1); }
        SB();
      }
      WKEEPA(ks);
    }
    STAMP(1);
    XPOLL();
    asm volatile("" ::: "memory");
#pragma unroll
    for (int i = 0; i < SW - 1; ++i) XLOAD(i, i);
    win[SW - 1] = u32x4{0u, 0u, 0u, 0u};
    WAIT1(SW - 2, win[0]);
    for (int it = 0; it < 8; ++it) {
      const unsigned qb = (unsigned)(it * SW + SW - 1) * 1024u;
#pragma unroll
      for (int h = 0; h < 2; ++h) {
        f32x4 acc[4];
#pragma unroll
        for (int k = h * KSV; k < (h + 1) * KSV; ++k) {
          if (k == h * KSV) STEP_(A, CIN_ZERO, "=&v", acc, SW - 2, k, (k + SW - 1) % SW, (k + 1) % SW, qb + (k >> 2) * 4096, (k & 3) * 1024, k - h * KSV);
          else STEP_(A, CIN_ACC, "+v", acc, SW - 2, k, (k + SW - 1) % SW, (k + 1) % SW, qb + (k >> 2) * 4096, (k & 3) * 1024, k - h * KSV);
        }
        MFMA_FENCE(acc);
        f32x4* pw = (f32x4*)(smem + LDS_OWN + (2 * it + h) * 4096 + xvo);
#pragma unroll
        for (int t = 0; t < 4; ++t) pw[t * 64] = acc[t];
        SB();
      }
    }
    asm volatile("s_waitcnt vmcnt(0)" ::: "memory");
#pragma unroll
    for (int i = 0; i < SW; ++i) asm volatile("" :: "v"(win[i]));
    STAMP(2);

    const bool local = dd == 0;
    char* dstp = local ? (char*)(P.out + (size_t)(P.rank * M_SH + rho * BM) * N_SH)
                       : P.ws[dest] + WS_MSG + (size_t)(((call & 1) * 8 + P.rank) * 16 + tile) * MSG_TILE;
    const __amdgpu_buffer_rsrc_t rxch = __builtin_amdgcn_make_buffer_rsrc((void*)(myws + WS_XCH + (size_t)pid * 65536), 0, 65536, 0x00020000);
    char* const buf = smem + LDS_STAGE + 3 * 5120;

    auto gather = [&](int o, f32x4 (&acc)[4]) {
      const f32x4* own = (const f32x4*)(smem + LDS_OWN + o * 4096 + xvo);
#pragma unroll
      for (int t = 0; t < 4; ++t) acc[t] = own[t * 64];
      while (cnts[o] < 3u) { if (TIMED_OUT()) { bad = true; break; } }
      asm volatile("" ::: "memory");
      const f32x4* part = (const f32x4*)(smem + (o % NRING) * RED_BYTES + xvo);
      f32x4 pr[12];
#pragma unroll
      for (int i = 0; i < 12; ++i) pr[i] = part[i * 64];
      SB();
#pragma unroll
      for (int t = 0; t < 4; ++t) {
        acc[t] = (acc[t] + pr[t]) + (pr[4 + t] + pr[8 + t]);
        asm volatile("" : "+v"(acc[t]) :: "memory");
      }
      if (lane == 0) cnts[16] = (unsigned)(o + 1);
      SB();
    };
    auto finalize = [&](int j, f32x4 (&a)[4], u32x4 (&pp)[4]) {
      const unsigned xo = (unsigned)j * 4096u;
      auto complete = [&]() {
        unsigned mn = ~0u;
#pragma unroll
        for (int t = 0; t < 4; ++t)
#pragma unroll
          for (int rg = 0; rg < 4; ++rg) mn = min(mn, pp[t][rg]);
        return __all(mn != 0u);
      };
      if (!complete()) {
        do {
          if (TIMED_OUT()) { bad = true; break; }
#pragma unroll
          for (int t = 0; t < 4; ++t) pp[t] = __builtin_amdgcn_raw_buffer_load_b128(rxch, xvo + t * 1024, xo, 16);
          asm volatile("" ::: "memory");
        } while (!complete());
      }
#pragma unroll
      for (int t = 0; t < 4; ++t) a[t] += __builtin_bit_cast(f32x4, pp[t]);
      if (bad) a[0][0] = __builtin_nanf("");
      if (local) {
        const __amdgpu_buffer_rsrc_t rd = __builtin_amdgcn_make_buffer_rsrc((void*)dstp, 0, BM * N_SH * 4, 0x00020000);
        const unsigned ovo = (xvo >> 8) * (4 * N_SH * 4) + (xvo & 255u);
        const unsigned so = (unsigned)(j * 16 * N_SH + strip * BN) * 4u;
#pragma unroll
        for (int rg = 0; rg < 4; ++rg) {
          f32x4 v;
#pragma unroll
          for (int t = 0; t < 4; ++t) v[t] = a[t][rg] <= 0.f ? 0.f : a[t][rg];
          __builtin_amdgcn_raw_buffer_store_b128(__builtin_bit_cast(u32x4, v), rd, ovo + rg * (N_SH * 4), so, 0);
        }
      } else {
        u64 bal[16];
        unsigned bits[16];
#pragma unroll
        for (int e = 0; e < 16; ++e) {
          const float av = a[e >> 2][e & 3];
          bal[e] = __builtin_amdgcn_ballot_w64(!(av <= 0.f));
          bits[e] = __builtin_bit_cast(unsigned, av) & 0x7fffffffu;
        }
        asm volatile("s_nop 1\n\tv_cndmask_b32 %0, %1, %0, %2" : "+v"(bits[15]) : "v"(0x80000000u), "s"(bal[15]));
        bal[15] |= 0xaaaaaaaaaaaaaaaaull;
        unsigned mask = 0;
        asm volatile("s_nop 1" ::: "memory");
#pragma unroll
        for (int e = 0; e < 16; ++e)
          asm volatile("v_addc_co_u32 %0, vcc, %0, %0, %1" : "+v"(mask) : "s"(bal[e]) : "vcc");
        const unsigned cnt = __builtin_popcount(mask);
        const unsigned incl = wave_scan(cnt);
        const unsigned n = __builtin_amdgcn_readlane(incl, 63);
        unsigned ad = (unsigned)(size_t)(LDS_AS char*)(buf + 128) + (incl - cnt) * 4u;
#pragma unroll
        for (int q = 0; q < 4; ++q)
          asm volatile("s_mov_b64 exec, %5\n\tds_write_b32 %0, %1\n\tv_add_u32 %0, 4, %0\n\t"
                       "s_mov_b64 exec, %6\n\tds_write_b32 %0, %2\n\tv_add_u32 %0, 4, %0\n\t"
                       "s_mov_b64 exec, %7\n\tds_write_b32 %0, %3\n\tv_add_u32 %0, 4, %0\n\t"
                       "s_mov_b64 exec, %8\n\tds_write_b32 %0, %4\n\tv_add_u32 %0, 4, %0\n\t"
                       "s_mov_b64 exec, -1"
                       : "+v"(ad) : "v"(bits[4 * q]), "v"(bits[4 * q + 1]), "v"(bits[4 * q + 2]), "v"(bits[4 * q + 3]),
                         "s"(bal[4 * q]), "s"(bal[4 * q + 1]), "s"(bal[4 * q + 2]), "s"(bal[4 * q + 3]) : "memory");
        *(unsigned short*)(buf + (xvo >> 3)) = (unsigned short)mask;
        const __amdgpu_buffer_rsrc_t rd = __builtin_amdgcn_make_buffer_rsrc((void*)(dstp + j * MSG_BYTES), 0, (int)((32u + n) * 4u), 0x00020000);
        asm volatile("s_waitcnt lgkmcnt(0)" ::: "memory");
        u32x4 v[NST];
#pragma unroll
        for (int s = 0; s < NST; ++s) v[s] = *(u32x4*)(buf + s * 1024 + xvo);
#pragma unroll
        for (int s = 0; s < NST; ++s) __builtin_amdgcn_raw_buffer_store_b128(v[s], rd, xvo + (s & 3) * 1024, (s >> 2) * 4096, 17);
      }
#pragma unroll
      for (int t = 0; t < 4; ++t) __builtin_amdgcn_raw_buffer_store_b128(u32x4{0u, 0u, 0u, 0u}, rxch, xvo + t * 1024, xo, 16);
      SB();
    };

    f32x4 held[4];
#pragma unroll
    for (int t = 0; t < 4; ++t) held[t] = f32x4{0.f, 0.f, 0.f, 0.f};
    for (int it = 0; it < 8; ++it) {
      const int jA = 2 * it + (kap ^ 1), jP = 2 * it - 2 + kap;
      u32x4 pp[4];
      const unsigned po = it > 0 ? (unsigned)jP * 4096u : OOB - 4096u;
#pragma unroll
      for (int t = 0; t < 4; ++t) pp[t] = __builtin_amdgcn_raw_buffer_load_b128(rxch, xvo + t * 1024, po, 16);
      asm volatile("" ::: "memory");
      f32x4 acc[4];
      gather(2 * it, acc);
#pragma unroll
      for (int t = 0; t < 4; ++t) {
        u32x4 v = __builtin_bit_cast(u32x4, acc[t]);
#pragma unroll
        for (int rg = 0; rg < 4; ++rg) v[rg] |= (v[rg] - 1u) & ~v[rg] & 0x80000000u;
        __builtin_amdgcn_raw_buffer_store_b128(v, rxch, xvo + t * 1024, (unsigned)jA * 4096u, 16);
      }
      SB();
      asm volatile("" : "+v"(pp[0]), "+v"(pp[1]), "+v"(pp[2]), "+v"(pp[3]));
      if (it > 0) finalize(jP, held, pp);
      if (it == 1) STAMP(3);
      gather(2 * it + 1, held);
    }
    STAMP(4);
    {
      u32x4 pp[4];
#pragma unroll
      for (int t = 0; t < 4; ++t) pp[t] = u32x4{0u, 0u, 0u, 0u};
      finalize(14 + kap, held, pp);
    }
    if (lane == 0) __hip_atomic_store((unsigned*)seqp, call, __ATOMIC_RELAXED, __HIP_MEMORY_SCOPE_AGENT);
    STAMP(5);
  }
  if (dd != 0 && wv < 3) {
    char* stage = smem + LDS_STAGE + wv * 5120;
    const char* mbase = myws + WS_MSG + (size_t)(((call & 1) * 8 + src) * 16 + tile) * MSG_TILE;
    const __amdgpu_buffer_rsrc_t rm = __builtin_amdgcn_make_buffer_rsrc((void*)mbase, 0, MSG_TILE, 0x00020000);
    const __amdgpu_buffer_rsrc_t ro = __builtin_amdgcn_make_buffer_rsrc((void*)(P.out + (size_t)(src * M_SH + rho * BM) * N_SH), 0, BM * N_SH * 4, 0x00020000);
    for (int m = wv; m < 8; m += 3) {
      const int j = 2 * m + kap;
      const unsigned mo = (unsigned)(j * MSG_BYTES);
      unsigned val[16];
      unsigned n;
      for (;;) {
        u32x4 mm[NST];
#pragma unroll
        for (int s = 0; s < NST; ++s) {
          const unsigned o = s * 1024 + lane * 16;
          mm[s] = __builtin_amdgcn_raw_buffer_load_b128(rm, o < MSG_BYTES ? mo + o : OOB, 0, 17);
        }
#pragma unroll
        for (int s = 0; s < NST; ++s) *(u32x4*)(stage + s * 1024 + lane * 16) = mm[s];
        asm volatile("s_waitcnt lgkmcnt(0)" ::: "memory");
        const unsigned mask = ((volatile unsigned short*)stage)[lane];
        const unsigned cnt = __builtin_popcount(mask);
        const unsigned incl = wave_scan(cnt);
        n = __builtin_amdgcn_readlane(incl, 63);
        unsigned ad = (unsigned)(size_t)(LDS_AS char*)(stage + 128) + (incl - cnt) * 4u;
        u64 bal[16];
        unsigned ms = mask << 16;
#pragma unroll
        for (int e = 0; e < 16; ++e) { asm volatile("v_add_co_u32 %0, %1, %0, %0" : "+v"(ms), "=s"(bal[e])); val[e] = 0x80000000u; }
#pragma unroll
        for (int q = 0; q < 4; ++q)
          asm volatile("s_mov_b64 exec, %5\n\tds_read_b32 %1, %0\n\tv_add_u32 %0, 4, %0\n\t"
                       "s_mov_b64 exec, %6\n\tds_read_b32 %2, %0\n\tv_add_u32 %0, 4, %0\n\t"
                       "s_mov_b64 exec, %7\n\tds_read_b32 %3, %0\n\tv_add_u32 %0, 4, %0\n\t"
                       "s_mov_b64 exec, %8\n\tds_read_b32 %4, %0\n\tv_add_u32 %0, 4, %0\n\t"
                       "s_mov_b64 exec, -1\n\ts_waitcnt lgkmcnt(0)"
                       : "+v"(ad), "+v"(val[4 * q]), "+v"(val[4 * q + 1]), "+v"(val[4 * q + 2]), "+v"(val[4 * q + 3])
                       : "s"(bal[4 * q]), "s"(bal[4 * q + 1]), "s"(bal[4 * q + 2]), "s"(bal[4 * q + 3]) : "memory");
        unsigned mn = (lane & 1) ? (mask & 1u) : 1u;
#pragma unroll
        for (int e = 0; e < 16; ++e) mn = min(mn, val[e]);
        if (__all(mn != 0u)) break;
        if (TIMED_OUT()) { bad = true; break; }
        __builtin_amdgcn_s_sleep(1);
      }
      const unsigned ovo = (unsigned)((j * 16 + 4 * g) * N_SH + strip * BN + 4 * c) * 4u;
#pragma unroll
      for (int rg = 0; rg < 4; ++rg) {
        u32x4 v;
#pragma unroll
        for (int t = 0; t < 4; ++t) v[t] = (unsigned)max((int)val[t * 4 + rg], 0);
        if (bad) v[0] = 0x7fc00000u;
        __builtin_amdgcn_raw_buffer_store_b128(v, ro, ovo + rg * (N_SH * 4), 0, 0);
      }
      const unsigned nb = (32u + n) * 4u;
#pragma unroll
      for (int s = 0; s < NST; ++s) {
        const unsigned o = s * 1024 + lane * 16;
        __builtin_amdgcn_raw_buffer_store_b128(u32x4{0u, 0u, 0u, 0u}, rm, o < nb ? mo + o : OOB, 0, 17);
      }
    }
  }
  STAMP(7);
  STAMP_FLUSH();
}

extern "C" void kernel_launch(int rank, int nranks, ncclComm_t comm, hipStream_t stream,
                              void* const* d_in, const int* in_sizes, int n_in,
                              void* d_out, int out_size, const int* in_dtypes, int out_dtype,
                              void* const* d_out_all, void* const* d_ws_all, size_t ws_bytes) {
  if (nranks != NRANK || (unsigned)rank >= (unsigned)NRANK || ws_bytes < WS_NEED) return;
  Params P;
  P.x = (const float*)d_in[0]; P.w = (const float*)d_in[1]; P.rank = rank; P.out = (float*)d_out;
  for (int p = 0; p < NRANK; ++p) P.ws[p] = (char*)d_ws_all[p];
  P.myws = P.ws[rank];
  HIP_CHECK(hipFuncSetAttribute((const void*)gemm_a2a, hipFuncAttributeMaxDynamicSharedMemorySize, LDS_BYTES));
  gemm_a2a<<<NWG, 256, LDS_BYTES, stream>>>(P);
  HIP_CHECK(hipGetLastError());
}
```

```cpp
#include <hip/hip_runtime.h>
#include <rccl/rccl.h>
#include <cstdio>

#define HIP_CHECK(expr) do { hipError_t _e = (expr); if (_e != hipSuccess) { \
    fprintf(stderr, "HIP error %s at %s:%d\n", hipGetErrorString(_e), __FILE__, __LINE__); \
    return; } } while (0)

typedef float f32x4 __attribute__((ext_vector_type(4)));
typedef _Float16 f16x2 __attribute__((ext_vector_type(2)));
typedef unsigned u32x4 __attribute__((ext_vector_type(4)));
typedef unsigned long long u64;
#define LDS_AS __attribute__((address_space(3)))

constexpr int NRANK = 8;
constexpr int M_SH = 512, K_DIM = 4096, N_DIM = 4096, N_SH = 512;
constexpr int BM = 256, BN = 64;
constexpr int NWG = 256;
#define EARLY 1
#define XW KC
#define SWR 8
#define KC 20
constexpr int KSV = 64 - 3 * KC;
constexpr int NST = 5;
constexpr u64 DEADLINE_TICKS = 50000000ull;
constexpr int WS_XFLAG = 4096;
constexpr int WS_SEQ = 49152;
constexpr int WS_DBG = 65536;
constexpr int WS_X16 = 1 << 20;
constexpr int WS_XCH = 10 << 20;
constexpr int MSG_BYTES = 4224;
constexpr int WS_MSG = 18 << 20;
constexpr int MSG_TILE = 16 * MSG_BYTES;
constexpr size_t WS_NEED = (size_t)WS_MSG + 2ull * 8 * 16 * MSG_TILE;
constexpr int RED_BYTES = 3 * 4096;
constexpr int NRING = 6;
constexpr int LDS_OWN = NRING * RED_BYTES;
constexpr int LDS_STAGE = LDS_OWN + 16 * 4096;
constexpr int LDS_CNT = LDS_STAGE + 4 * 5120;
constexpr int LDS_BYTES = LDS_CNT + 256;
constexpr unsigned OOB = 0x7ffffff0u;

struct Params {
  const float* x; const float* w;
  float* out;
  char* myws;
  char* ws[NRANK];
  int rank;
};

__device__ __forceinline__ unsigned pk(float a, float b) {
  f16x2 r; r[0] = (_Float16)a; r[1] = (_Float16)b; return __builtin_bit_cast(unsigned, r);
}
__device__ __forceinline__ u32x4 make_srd(const void* p, unsigned bytes) {
  const u64 a = (u64)p;
  u32x4 r;
  r[0] = __builtin_amdgcn_readfirstlane((unsigned)a);
  r[1] = __builtin_amdgcn_readfirstlane((unsigned)(a >> 32)) & 0xffffu;
  r[2] = bytes; r[3] = 0x00020000u;
  return r;
}
__device__ __forceinline__ unsigned wave_scan(unsigned v) {
  v += __builtin_amdgcn_update_dpp(0u, v, 0x111, 0xf, 0xf, false);
  v += __builtin_amdgcn_update_dpp(0u, v, 0x112, 0xf, 0xf, false);
  v += __builtin_amdgcn_update_dpp(0u, v, 0x114, 0xf, 0xf, false);
  v += __builtin_amdgcn_update_dpp(0u, v, 0x118, 0xf, 0xf, false);
  v += __builtin_amdgcn_update_dpp(0u, v, 0x142, 0xa, 0xf, false);
  v += __builtin_amdgcn_update_dpp(0u, v, 0x143, 0xc, 0xf, false);
  return v;
}
#define STAMP(i)
#define STAMP_FLUSH()
#define STAMP_DECL
#define SB() __builtin_amdgcn_sched_barrier(0)
#define TIMED_OUT() (__builtin_amdgcn_s_memrealtime() - t_start > DEADLINE_TICKS)

__global__ void __launch_bounds__(256, 1) gemm_a2a(Params P) {
  extern __shared__ __attribute__((aligned(16))) char smem[];
  const int tid = threadIdx.x, lane = tid & 63, wv = __builtin_amdgcn_readfirstlane(tid >> 6);
  const int bid = blockIdx.x;
  const int idx = bid >> 3;
  const int kap = bid & 1, rho = idx & 1, dd = idx >> 2;
  const int strip = (bid & 6) + ((idx >> 1) & 1);
  const int dest = (P.rank + dd) & 7, src = (P.rank - dd) & 7;
  const int tile = rho * 8 + strip, pid = dd * 16 + tile;
  char* myws = P.myws;
  const int c = lane & 15, g = lane >> 4;
  STAMP_DECL
  STAMP(0);
  const u64 t_start = __builtin_amdgcn_s_memrealtime();
  volatile LDS_AS unsigned* cnts = (volatile LDS_AS unsigned*)(smem + LDS_CNT);
  const char* seqp = myws + WS_SEQ + 64 * bid;
  unsigned call;
  if (tid < 32) ((unsigned*)(smem + LDS_CNT))[tid] = 0u;

  const u32x4 rw = make_srd(P.w, K_DIM * N_DIM * 4);
  u32x4 rxs;
#define BLOAD(dst, vo, rs, so, imm, aux) asm volatile("buffer_load_dwordx4 %0, %1, %2, %3 offen offset:%4" aux : "=v"(dst) : "v"(vo), "s"(rs), "s"(so), "n"(imm) : "memory")
#define BSTORE(v, vo, rs, so, imm, aux) asm volatile("buffer_store_dwordx4 %0, %1, %2, %3 offen offset:%4" aux "\n\ts_nop 1" :: "v"(v), "v"(vo), "s"(rs), "s"(so), "n"(imm) : "memory")
#define WAIT1(n, a) asm volatile("s_waitcnt vmcnt(%1)" : "+v"(a) : "n"(n) : "memory")
#define WAIT2(n, a, b) asm volatile("s_waitcnt vmcnt(%2)" : "+v"(a), "+v"(b) : "n"(n) : "memory")
  const unsigned wvo = (unsigned)(8 * g * N_DIM + dest * N_SH + strip * BN + 4 * c) * 4u;
  const unsigned wso = (unsigned)(kap * 2048 + (wv < 3 ? wv : 3 * KC) * 32) * (N_DIM * 4u);
  const unsigned wsoA = wso + rho * (4 * N_DIM * 4u), wsoB = wso - rho * (4 * N_DIM * 4u);
  const unsigned xvo = lane * 16;
  bool bad = false;
  unsigned* const xflag = (unsigned*)(myws + WS_XFLAG + 8192 * (rho * 2 + kap) + 128 * lane);
#define WAUX ""
#define WLOAD(i) BLOAD(wl[i], wvo, rw, (((i) & 4) ? wsoB : wsoA) + (unsigned)(((i) >> 3) * WKS * 32 + ((i) & 7)) * (N_DIM * 4u), 0, WAUX)
#define XLOAD(s, q) BLOAD(win[s], xvo, rxs, (unsigned)(((q) >> 2) * 4096), ((q) & 3) * 1024, " sc1")
#define WCVT(i, jj) do { _Pragma("unroll") for (int t = 0; t < 4; ++t) { bb[t][jj] = pk(wl[i][t], wl[(i) + 1][t]); asm volatile("" : "+v"(bb[t][jj])); } SB(); } while (0)
#define WKEEP_(C, ks) do { _Pragma("unroll") for (int t = 0; t < 4; ++t) { u32x4 b_ = {bb[t][0], bb[t][1], bb[t][2], bb[t][3]}; \
    asm volatile("" : C(b_)); breg[ks][t] = b_; } SB(); } while (0)
#define WKEEPA(ks) WKEEP_("+a", ks)
#define WKEEPV(ks) WKEEP_("+v", ks)
#define GETCALL() do { asm volatile("s_load_dword %0, %1, 0x0\n\ts_waitcnt lgkmcnt(0)" : "=s"(call) : "s"(seqp) : "memory"); call += 1u; \
    rxs = make_srd(myws + WS_X16 + (size_t)(call & 1) * (M_SH * K_DIM * 2) + (rho * 2 + kap) * 1048576 + wv * (16 * KC * 1024), \
                   (wv < 3 ? 16 * KC : 16 * KSV) * 1024u);                                                           \
    __builtin_amdgcn_s_barrier();                                                            \
    asm volatile("s_nop 4" ::: "memory"); } while (0)
#define XPOLL() do { while (!__all(__hip_atomic_load(xflag, __ATOMIC_RELAXED, __HIP_MEMORY_SCOPE_AGENT) >= call)) {   \
      if (TIMED_OUT()) { bad = true; break; }                                                                        \
      __builtin_amdgcn_s_sleep(4); } } while (0)
#define MF(i, CIN) "v_mfma_f32_16x16x32_f16 %[c" #i "], %[a], %[b" #i "], " CIN(i) "\n\t"
#define CIN_ACC(i) "%[c" #i "]"
#define CIN_ZERO(i) "0"
#define COUT(CC, A) [c0] CC(A[0]), [c1] CC(A[1]), [c2] CC(A[2]), [c3] CC(A[3])
#define BC_A "a"
#define BC_V "v"
#define BIN(K, ks) [b0] BC_##K(breg[ks][0]), [b1] BC_##K(breg[ks][1]), [b2] BC_##K(breg[ks][2]), [b3] BC_##K(breg[ks][3])
#define STEP_(K, CIN, CC, A, n_, s, sp, sn, so_, imm_, ks) do { asm volatile(MF(0, CIN)                               \
    "buffer_load_dwordx4 %[p], %[vo], %[rs], %[so] offen offset:%[imm] sc1\n\t" MF(1, CIN) MF(2, CIN)                \
    "s_waitcnt vmcnt(%[n])\n\t" MF(3, CIN)                                                                           \
    : COUT(CC, A), [p] "+v"(win[sp]), [nx] "+v"(win[sn])                                                             \
    : [a] "v"(win[s]), BIN(K, ks), [vo] "v"(xvo), [rs] "s"(rxs), [so] "s"(so_), [imm] "n"(imm_), [n] "n"(n_) : "memory"); SB(); } while (0)
#define MFMA_FENCE(A) asm volatile("s_nop 7\n\ts_nop 1" : "+v"(A[0]), "+v"(A[1]), "+v"(A[2]), "+v"(A[3]))

  if (wv < 3) {
    constexpr int WKS = 3;
    f32x4 wl[8 * KC];
    u32x4 win[XW];
    u32x4 breg[KC][4];
    f32x4 eacc[EARLY][4];
    unsigned xc = 0;
#define P1_INIT() GETCALL()
#define P1_CLOAD() do { asm volatile("global_load_dword %0, %1, off sc1" : "=v"(xc) : "v"(xflag) : "memory"); SB(); } while (0)
#define P1_CHECK() do { asm volatile("" : "+v"(xc)); if (!__all(xc >= call)) XPOLL(); asm volatile("" ::: "memory");  \
    if (tid == 0) cnts[17] = 1u; SB(); STAMP(1); } while (0)
#define EMFMA_(K, CIN, CC, n_, s, e, ks) do { asm volatile("s_waitcnt vmcnt(%[n])\n\ts_nop 0\n\t" MF(0, CIN) MF(1, CIN) MF(2, CIN) MF(3, CIN) \
    : COUT(CC, eacc[e]), [a] "+v"(win[s]) : BIN(K, ks), [n] "n"(n_) : "memory"); SB(); } while (0)
#define EMFMA0(K, n_, s, e, ks) EMFMA_(K, CIN_ZERO, "=&v", n_, s, e, ks)
#define EMFMA(K, n_, s, e, ks) EMFMA_(K, CIN_ACC, "+v", n_, s, e, ks)
#define ESTEP(K, n_, s, sp, sn, q, e, ks) STEP_(K, CIN_ACC, "+v", eacc[e], n_, s, sp, sn, (unsigned)(((q) >> 2) * 4096), ((q) & 3) * 1024, ks)
    asm volatile("s_nop 4" ::: "memory");
    WLOAD(0);
    WLOAD(1);
    WLOAD(2);
    WLOAD(3);
    WLOAD(4);
    WLOAD(5);
    WLOAD(6);
    WLOAD(7);
    SB();
    P1_INIT();
    { unsigned bb[4][4];
    WAIT2(6, wl[0], wl[1]);
    WCVT(0, 0);
    WLOAD(8);
    WLOAD(9);
    SB();
    WAIT2(6, wl[2], wl[3]);
    WCVT(2, 1);
    WLOAD(10);
    WLOAD(11);
    SB();
    WAIT2(6, wl[4], wl[5]);
    WCVT(4, 2);
    WLOAD(12);
    WLOAD(13);
    SB();
    WAIT2(6, wl[6], wl[7]);
    WCVT(6, 3);
    WLOAD(14);
    WLOAD(15);
    SB();
    WKEEPA(0); }
    { unsigned bb[4][4];
    WAIT2(6, wl[8], wl[9]);
    WCVT(8, 0);
    WLOAD(16);
    WLOAD(17);
    SB();
    WAIT2(6, wl[10], wl[11]);
    WCVT(10, 1);
    WLOAD(18);
    WLOAD(19);
    SB();
    WAIT2(6, wl[12], wl[13]);
    WCVT(12, 2);
    WLOAD(20);
    WLOAD(21);
    SB();
    WAIT2(6, wl[14], wl[15]);
    WCVT(14, 3);
    WLOAD(22);
    WLOAD(23);
    SB();
    WKEEPA(1); }
    { unsigned bb[4][4];
    WAIT2(6, wl[16], wl[17]);
    WCVT(16, 0);
    WLOAD(24);
    WLOAD(25);
    SB();
    WAIT2(6, wl[18], wl[19]);
    WCVT(18, 1);
    WLOAD(26);
    WLOAD(27);
    SB();
    WAIT2(6, wl[20], wl[21]);
    WCVT(20, 2);
    WLOAD(28);
    WLOAD(29);
    SB();
    WAIT2(6, wl[22], wl[23]);
    WCVT(22, 3);
    WLOAD(30);
    WLOAD(31);
    SB();
    WKEEPA(2); }
    { unsigned bb[4][4];
    WAIT2(6, wl[24], wl[25]);
    WCVT(24, 0);
    WLOAD(32);
    WLOAD(33);
    SB();
    WAIT2(6, wl[26], wl[27]);
    WCVT(26, 1);
    WLOAD(34);
    WLOAD(35);
    SB();
    WAIT2(6, wl[28], wl[29]);
    WCVT(28, 2);
    WLOAD(36);
    WLOAD(37);
    SB();
    WAIT2(6, wl[30], wl[31]);
    WCVT(30, 3);
    WLOAD(38);
    WLOAD(39);
    SB();
    WKEEPA(3); }
    { unsigned bb[4][4];
    WAIT2(6, wl[32], wl[33]);
    WCVT(32, 0);
    WLOAD(40);
    WLOAD(41);
    SB();
    WAIT2(6, wl[34], wl[35]);
    WCVT(34, 1);
    WLOAD(42);
    WLOAD(43);
    SB();
    WAIT2(6, wl[36], wl[37]);
    WCVT(36, 2);
    WLOAD(44);
    WLOAD(45);
    SB();
    WAIT2(6, wl[38], wl[39]);
    WCVT(38, 3);
    WLOAD(46);
    WLOAD(47);
    SB();
    WKEEPA(4); }
    { unsigned bb[4][4];
    WAIT2(6, wl[40], wl[41]);
    WCVT(40, 0);
    WLOAD(48);
    WLOAD(49);
    SB();
    WAIT2(6, wl[42], wl[43]);
    WCVT(42, 1);
    WLOAD(50);
    WLOAD(51);
    SB();
    WAIT2(6, wl[44], wl[45]);
    WCVT(44, 2);
    WLOAD(52);
    WLOAD(53);
    SB();
    WAIT2(6, wl[46], wl[47]);
    WCVT(46, 3);
    WLOAD(54);
    WLOAD(55);
    SB();
    WKEEPA(5); }
    { unsigned bb[4][4];
    WAIT2(6, wl[48], wl[49]);
    WCVT(48, 0);
    WLOAD(56);
    WLOAD(57);
    SB();
    WAIT2(6, wl[50], wl[51]);
    WCVT(50, 1);
    WLOAD(58);
    WLOAD(59);
    SB();
    WAIT2(6, wl[52], wl[53]);
    WCVT(52, 2);
    WLOAD(60);
    WLOAD(61);
    SB();
    WAIT2(6, wl[54], wl[55]);
    WCVT(54, 3);
    WLOAD(62);
    WLOAD(63);
    SB();
    WKEEPA(6); }
    { unsigned bb[4][4];
    WAIT2(6, wl[56], wl[57]);
    WCVT(56, 0);
    WLOAD(64);
    WLOAD(65);
    SB();
    WAIT2(6, wl[58], wl[59]);
    WCVT(58, 1);
    WLOAD(66);
    WLOAD(67);
    SB();
    WAIT2(6, wl[60], wl[61]);
    WCVT(60, 2);
    WLOAD(68);
    WLOAD(69);
    SB();
    WAIT2(6, wl[62], wl[63]);
    WCVT(62, 3);
    WLOAD(70);
    WLOAD(71);
    SB();
    WKEEPA(7); }
    { unsigned bb[4][4];
    WAIT2(6, wl[64], wl[65]);
    WCVT(64, 0);
    WLOAD(72);
    WLOAD(73);
    SB();
    WAIT2(6, wl[66], wl[67]);
    WCVT(66, 1);
    WLOAD(74);
    WLOAD(75);
    SB();
    WAIT2(6, wl[68], wl[69]);
    WCVT(68, 2);
    WLOAD(76);
    WLOAD(77);
    SB();
    WAIT2(6, wl[70], wl[71]);
    WCVT(70, 3);
    WLOAD(78);
    WLOAD(79);
    SB();
    WKEEPA(8); }
    { unsigned bb[4][4];
    WAIT2(6, wl[72], wl[73]);
    WCVT(72, 0);
    WLOAD(80);
    WLOAD(81);
    SB();
    WAIT2(6, wl[74], wl[75]);
    WCVT(74, 1);
    WLOAD(82);
    WLOAD(83);
    SB();
    WAIT2(6, wl[76], wl[77]);
    WCVT(76, 2);
    WLOAD(84);
    WLOAD(85);
    SB();
    WAIT2(6, wl[78], wl[79]);
    WCVT(78, 3);
    WLOAD(86);
    WLOAD(87);
    SB();
    WKEEPA(9); }
    { unsigned bb[4][4];
    WAIT2(6, wl[80], wl[81]);
    WCVT(80, 0);
    WLOAD(88);
    WLOAD(89);
    SB();
    WAIT2(6, wl[82], wl[83]);
    WCVT(82, 1);
    WLOAD(90);
    WLOAD(91);
    SB();
    WAIT2(6, wl[84], wl[85]);
    WCVT(84, 2);
    WLOAD(92);
    WLOAD(93);
    SB();
    WAIT2(6, wl[86], wl[87]);
    WCVT(86, 3);
    WLOAD(94);
    WLOAD(95);
    SB();
    WKEEPA(10); }
    { unsigned bb[4][4];
    WAIT2(6, wl[88], wl[89]);
    WCVT(88, 0);
    WLOAD(96);
    WLOAD(97);
    SB();
    WAIT2(6, wl[90], wl[91]);
    WCVT(90, 1);
    WLOAD(98);
    WLOAD(99);
    SB();
    WAIT2(6, wl[92], wl[93]);
    WCVT(92, 2);
    WLOAD(100);
    WLOAD(101);
    SB();
    WAIT2(6, wl[94], wl[95]);
    WCVT(94, 3);
    WLOAD(102);
    WLOAD(103);
    SB();
    WKEEPA(11); }
    { unsigned bb[4][4];
    WAIT2(6, wl[96], wl[97]);
    WCVT(96, 0);
    WLOAD(104);
    WLOAD(105);
    SB();
    WAIT2(6, wl[98], wl[99]);
    WCVT(98, 1);
    WLOAD(106);
    WLOAD(107);
    SB();
    WAIT2(6, wl[100], wl[101]);
    WCVT(100, 2);
    WLOAD(108);
    WLOAD(109);
    SB();
    WAIT2(6, wl[102], wl[103]);
    WCVT(102, 3);
    WLOAD(110);
    WLOAD(111);
    SB();
    WKEEPA(12); }
    { unsigned bb[4][4];
    WAIT2(6, wl[104], wl[105]);
    WCVT(104, 0);
    WLOAD(112);
    WLOAD(113);
    SB();
    WAIT2(6, wl[106], wl[107]);
    WCVT(106, 1);
    WLOAD(114);
    WLOAD(115);
    SB();
    WAIT2(6, wl[108], wl[109]);
    WCVT(108, 2);
    WLOAD(116);
    WLOAD(117);
    SB();
    WAIT2(6, wl[110], wl[111]);
    WCVT(110, 3);
    WLOAD(118);
    WLOAD(119);
    SB();
    WKEEPA(13); }
    { unsigned bb[4][4];
    WAIT2(6, wl[112], wl[113]);
    WCVT(112, 0);
    WLOAD(120);
    WLOAD(121);
    SB();
    WAIT2(6, wl[114], wl[115]);
    WCVT(114, 1);
    WLOAD(122);
    WLOAD(123);
    SB();
    WAIT2(6, wl[116], wl[117]);
    WCVT(116, 2);
    WLOAD(124);
    WLOAD(125);
    SB();
    WAIT2(6, wl[118], wl[119]);
    WCVT(118, 3);
    WLOAD(126);
    WLOAD(127);
    SB();
    WKEEPA(14); }
    { unsigned bb[4][4];
    WAIT2(6, wl[120], wl[121]);
    WCVT(120, 0);
    WLOAD(128);
    WLOAD(129);
    SB();
    WAIT2(6, wl[122], wl[123]);
    WCVT(122, 1);
    WLOAD(130);
    WLOAD(131);
    SB();
    WAIT2(6, wl[124], wl[125]);
    WCVT(124, 2);
    WLOAD(132);
    WLOAD(133);
    SB();
    WAIT2(6, wl[126], wl[127]);
    WCVT(126, 3);
    WLOAD(134);
    WLOAD(135);
    SB();
    WKEEPA(15); }
    { unsigned bb[4][4];
    WAIT2(6, wl[128], wl[129]);
    WCVT(128, 0);
    WLOAD(136);
    WLOAD(137);
    SB();
    WAIT2(6, wl[130], wl[131]);
    WCVT(130, 1);
    WLOAD(138);
    WLOAD(139);
    SB();
    WAIT2(6, wl[132], wl[133]);
    WCVT(132, 2);
    WLOAD(140);
    WLOAD(141);
    SB();
    P1_CLOAD();
    WAIT2(7, wl[134], wl[135]);
    WCVT(134, 3);
    WLOAD(142);
    WLOAD(143);
    SB();
    WKEEPV(16); }
    { unsigned bb[4][4];
    WAIT2(7, wl[136], wl[137]);
    WCVT(136, 0);
    WLOAD(144);
    WLOAD(145);
    SB();
    WAIT2(7, wl[138], wl[139]);
    WCVT(138, 1);
    WLOAD(146);
    WLOAD(147);
    SB();
    WAIT2(7, wl[140], wl[141]);
    WCVT(140, 2);
    WLOAD(148);
    WLOAD(149);
    SB();
    WAIT2(6, wl[142], wl[143]);
    WCVT(142, 3);
    WLOAD(150);
    WLOAD(151);
    SB();
    WKEEPV(17); }
    P1_CHECK();
    XLOAD(0, 0);
    XLOAD(1, 1);
    { unsigned bb[4][4];
    WAIT2(8, wl[144], wl[145]);
    WCVT(144, 0);
    WLOAD(152);
    WLOAD(153);
    SB();
    XLOAD(2, 2);
    XLOAD(3, 3);
    WAIT2(10, wl[146], wl[147]);
    WCVT(146, 1);
    WLOAD(154);
    WLOAD(155);
    SB();
    XLOAD(4, 4);
    XLOAD(5, 5);
    WAIT2(12, wl[148], wl[149]);
    WCVT(148, 2);
    WLOAD(156);
    WLOAD(157);
    SB();
    XLOAD(6, 6);
    XLOAD(7, 7);
    WAIT2(14, wl[150], wl[151]);
    WCVT(150, 3);
    WLOAD(158);
    WLOAD(159);
    SB();
    WKEEPV(18); }
    EMFMA0(A, 15, 0, 0, 0);
    EMFMA(A, 14, 1, 0, 1);
    XLOAD(8, 8);
    XLOAD(9, 9);
    { unsigned bb[4][4];
    WAIT2(14, wl[152], wl[153]);
    WCVT(152, 0);
    SB();
    EMFMA(A, 13, 2, 0, 2);
    EMFMA(A, 12, 3, 0, 3);
    XLOAD(10, 10);
    XLOAD(11, 11);
    WAIT2(12, wl[154], wl[155]);
    WCVT(154, 1);
    SB();
    EMFMA(A, 11, 4, 0, 4);
    EMFMA(A, 10, 5, 0, 5);
    XLOAD(12, 12);
    XLOAD(13, 13);
    WAIT2(10, wl[156], wl[157]);
    WCVT(156, 2);
    SB();
    EMFMA(A, 9, 6, 0, 6);
    EMFMA(A, 8, 7, 0, 7);
    XLOAD(14, 14);
    XLOAD(15, 15);
    WAIT2(8, wl[158], wl[159]);
    WCVT(158, 3);
    SB();
    WKEEPV(19); }
    EMFMA(A, 7, 8, 0, 8);
    EMFMA(A, 6, 9, 0, 9);
    XLOAD(16, 16);
    XLOAD(17, 17);
    STAMP(3);
    EMFMA(A, 7, 10, 0, 10);
    XLOAD(18, 18);
    XLOAD(19, 19);
    EMFMA(A, 8, 11, 0, 11);
    XLOAD(0, 20);
    XLOAD(1, 21);
    EMFMA(A, 9, 12, 0, 12);
    XLOAD(2, 22);
    XLOAD(3, 23);
    EMFMA(A, 10, 13, 0, 13);
    XLOAD(4, 24);
    XLOAD(5, 25);
    EMFMA(A, 11, 14, 0, 14);
    XLOAD(6, 26);
    XLOAD(7, 27);
    EMFMA(A, 12, 15, 0, 15);
    XLOAD(8, 28);
    XLOAD(9, 29);
    EMFMA(V, 13, 16, 0, 16);
    XLOAD(10, 30);
    XLOAD(11, 31);
    EMFMA(V, 14, 17, 0, 17);
    XLOAD(12, 32);
    XLOAD(13, 33);
    EMFMA(V, 15, 18, 0, 18);
    XLOAD(14, 34);
    XLOAD(15, 35);
    XLOAD(16, 36);
    XLOAD(17, 37);
    WAIT1(18, win[19]);
    asm volatile("s_nop 1" ::: "memory");
    ESTEP(V, 18, 19, 18, 0, 38, 0, 19);
    STAMP(2);
    auto put_partial = [&](int o, const f32x4 (&acc)[4]) {
      f32x4* pw = (f32x4*)(smem + (o % NRING) * RED_BYTES + wv * 4096 + xvo);
#pragma unroll
      for (int t = 0; t < 4; ++t) pw[t * 64] = acc[t];
      asm volatile("" ::: "memory");
      if (lane == 0) asm volatile("ds_add_u32 %0, %1" :: "v"((unsigned)(size_t)(LDS_AS char*)(smem + LDS_CNT) + o * 4u), "v"(1u) : "memory");
      SB();
    };
    MFMA_FENCE(eacc[EARLY - 1]);
#pragma unroll
    for (int e = 0; e < EARLY; ++e) put_partial(e, eacc[e]);
    for (int o = EARLY; o < 16; ++o) {
      const unsigned qb = (unsigned)(o * KC + XW - 1) * 1024u;
      f32x4 acc[4];
      const unsigned consumed = cnts[16];
#pragma unroll
      for (int k = 0; k < KC; ++k) {
        if (k == 0) STEP_(A, CIN_ZERO, "=&v", acc, XW - 2, k % XW, (k + XW - 1) % XW, (k + 1) % XW, qb + (k >> 2) * 4096, (k & 3) * 1024, k);
        else if (k < 16) STEP_(A, CIN_ACC, "+v", acc, XW - 2, k % XW, (k + XW - 1) % XW, (k + 1) % XW, qb + (k >> 2) * 4096, (k & 3) * 1024, k);
        else STEP_(V, CIN_ACC, "+v", acc, XW - 2, k % XW, (k + XW - 1) % XW, (k + 1) % XW, qb + (k >> 2) * 4096, (k & 3) * 1024, k);
      }
      MFMA_FENCE(acc);
      if ((int)__builtin_amdgcn_readfirstlane(consumed) < o - (NRING - 1)) {
        while ((int)cnts[16] < o - (NRING - 1)) { if (TIMED_OUT()) { bad = true; break; } }
      }
      asm volatile("" ::: "memory");
      put_partial(o, acc);
    }
    STAMP(4);
    asm volatile("s_waitcnt vmcnt(0)" ::: "memory");
#pragma unroll
    for (int i = 0; i < XW; ++i) asm volatile("" :: "v"(win[i]));
  } else {
    constexpr int WKS = 1;
    constexpr int SW = 2 * KSV;
    f32x4 wl[8 * KSV];
    u32x4 win[SW];
    u32x4 breg[KSV][4];
    {
      const int rc = bid >> 3, kb = bid & 7;
      const u32x4 rx = make_srd(P.x, M_SH * K_DIM * 4);
      const unsigned xcvo = (unsigned)((rc * 16 + c) * K_DIM + kb * 512 + g * 8) * 4u;
      const unsigned xcvA = xcvo + (rc >> 4) * 16u, xcvB = xcvo + 16u - (rc >> 4) * 16u;
      f32x4 xl[32];
      asm volatile("s_nop 4" ::: "memory");
#pragma unroll
      for (int i = 0; i < 32; ++i) BLOAD(xl[i], (i & 1) ? xcvB : xcvA, rx, 0u, (i >> 1) * 128, "");
      GETCALL();
      const int grp = (rc >> 4) * 2 + (kb >> 2), oc = (rc & 15) ^ ((kb >> 2) ^ 1);
      const u32x4 rx16w = make_srd(myws + WS_X16 + (size_t)(call & 1) * (M_SH * K_DIM * 2) + grp * 1048576, 1048576u);
      asm volatile("s_nop 4" ::: "memory");
#pragma unroll
      for (int i = 0; i < 16; ++i) {
        const int gk = (kb & 3) * 16 + i;
        const int u = gk >= 3 * KC ? 3 : gk % 3;
        const int ks = gk >= 3 * KC ? gk - 3 * KC : gk / 3;
        const int pos = u * 16 * KC + (u == 3 ? oc * KSV + ks : oc < EARLY ? ks * EARLY + oc : oc * KC + ks);
        WAIT2(30 - i, xl[2 * i], xl[2 * i + 1]);
        const f32x4 a_ = xl[2 * i], b_ = xl[2 * i + 1];
        u32x4 o_ = {pk(a_[0], a_[1]), pk(a_[2], a_[3]), pk(b_[0], b_[1]), pk(b_[2], b_[3])};
        BSTORE(o_, xvo, rx16w, (unsigned)pos * 1024u, 0, " sc1");
      }
      SB();
#pragma unroll
      for (int i = 0; i < SWR; ++i) WLOAD(i);
      asm volatile("s_waitcnt vmcnt(%0)" :: "n"(SWR) : "memory");
      if (lane == 0) {
        unsigned* f_ = (unsigned*)(myws + WS_XFLAG + 8192 * grp + 128 * ((rc & 15) * 4 + (kb & 3)));
        asm volatile("global_store_dword %0, %1, off sc1" :: "v"(f_), "v"(call) : "memory");
      }
      SB();
      STAMP(6);
    }
#pragma unroll
    for (int ks = 0; ks < KSV; ++ks) {
      unsigned bb[4][4];
#pragma unroll
      for (int jj = 0; jj < 4; ++jj) {
        const int i = ks * 8 + 2 * jj;
        WAIT2(i + SWR <= 8 * KSV ? SWR - 2 : 8 * KSV - 2 - i, wl[i], wl[i + 1]);
        WCVT(i, jj);
        if (i + SWR < 8 * KSV) { WLOAD(i + SWR); WLOAD(i + SWR + 1); }
        SB();
      }
      WKEEPA(ks);
    }
    STAMP(1);
    while (cnts[17] == 0u) { if (TIMED_OUT()) { bad = true; break; } __builtin_amdgcn_s_sleep(2); }
    asm volatile("" ::: "memory");
#pragma unroll
    for (int i = 0; i < SW - 1; ++i) XLOAD(i, i);
    win[SW - 1] = u32x4{0u, 0u, 0u, 0u};
    WAIT1(SW - 2, win[0]);
    for (int it = 0; it < 8; ++it) {
      const unsigned qb = (unsigned)(it * SW + SW - 1) * 1024u;
#pragma unroll
      for (int h = 0; h < 2; ++h) {
        f32x4 acc[4];
#pragma unroll
        for (int k = h * KSV; k < (h + 1) * KSV; ++k) {
          if (k == h * KSV) STEP_(A, CIN_ZERO, "=&v", acc, SW - 2, k, (k + SW - 1) % SW, (k + 1) % SW, qb + (k >> 2) * 4096, (k & 3) * 1024, k - h * KSV);
          else STEP_(A, CIN_ACC, "+v", acc, SW - 2, k, (k + SW - 1) % SW, (k + 1) % SW, qb + (k >> 2) * 4096, (k & 3) * 1024, k - h * KSV);
        }
        MFMA_FENCE(acc);
        f32x4* pw = (f32x4*)(smem + LDS_OWN + (2 * it + h) * 4096 + xvo);
#pragma unroll
        for (int t = 0; t < 4; ++t) pw[t * 64] = acc[t];
        SB();
      }
    }
    asm volatile("s_waitcnt vmcnt(0)" ::: "memory");
#pragma unroll
    for (int i = 0; i < SW; ++i) asm volatile("" :: "v"(win[i]));
    STAMP(2);

    const bool local = dd == 0;
    char* dstp = local ? (char*)(P.out + (size_t)(P.rank * M_SH + rho * BM) * N_SH)
                       : P.ws[dest] + WS_MSG + (size_t)(((call & 1) * 8 + P.rank) * 16 + tile) * MSG_TILE;
    const __amdgpu_buffer_rsrc_t rxch = __builtin_amdgcn_make_buffer_rsrc((void*)(myws + WS_XCH + (size_t)pid * 65536), 0, 65536, 0x00020000);
    char* const buf = smem + LDS_STAGE + 3 * 5120;

    auto gather = [&](int o, f32x4 (&acc)[4]) {
      const f32x4* own = (const f32x4*)(smem + LDS_OWN + o * 4096 + xvo);
#pragma unroll
      for (int t = 0; t < 4; ++t) acc[t] = own[t * 64];
      while (cnts[o] < 3u) { if (TIMED_OUT()) { bad = true; break; } }
      asm volatile("" ::: "memory");
      const f32x4* part = (const f32x4*)(smem + (o % NRING) * RED_BYTES + xvo);
      f32x4 pr[12];
#pragma unroll
      for (int i = 0; i < 12; ++i) pr[i] = part[i * 64];
      SB();
#pragma unroll
      for (int t = 0; t < 4; ++t) {
        acc[t] = (acc[t] + pr[t]) + (pr[4 + t] + pr[8 + t]);
        asm volatile("" : "+v"(acc[t]) :: "memory");
      }
      if (lane == 0) cnts[16] = (unsigned)(o + 1);
      SB();
    };
    auto finalize = [&](int j, f32x4 (&a)[4], u32x4 (&pp)[4]) {
      const unsigned xo = (unsigned)j * 4096u;
      auto complete = [&]() {
        unsigned mn = ~0u;
#pragma unroll
        for (int t = 0; t < 4; ++t)
#pragma unroll
          for (int rg = 0; rg < 4; ++rg) mn = min(mn, pp[t][rg]);
        return __all(mn != 0u);
      };
      if (!complete()) {
        do {
          if (TIMED_OUT()) { bad = true; break; }
#pragma unroll
          for (int t = 0; t < 4; ++t) pp[t] = __builtin_amdgcn_raw_buffer_load_b128(rxch, xvo + t * 1024, xo, 16);
          asm volatile("" ::: "memory");
        } while (!complete());
      }
#pragma unroll
      for (int t = 0; t < 4; ++t) a[t] += __builtin_bit_cast(f32x4, pp[t]);
      if (bad) a[0][0] = __builtin_nanf("");
      if (local) {
        const __amdgpu_buffer_rsrc_t rd = __builtin_amdgcn_make_buffer_rsrc((void*)dstp, 0, BM * N_SH * 4, 0x00020000);
        const unsigned ovo = (xvo >> 8) * (4 * N_SH * 4) + (xvo & 255u);
        const unsigned so = (unsigned)(j * 16 * N_SH + strip * BN) * 4u;
#pragma unroll
        for (int rg = 0; rg < 4; ++rg) {
          f32x4 v;
#pragma unroll
          for (int t = 0; t < 4; ++t) v[t] = a[t][rg] <= 0.f ? 0.f : a[t][rg];
          __builtin_amdgcn_raw_buffer_store_b128(__builtin_bit_cast(u32x4, v), rd, ovo + so + rg * (N_SH * 4), 0, 0);
        }
      } else {
        u64 bal[16];
        unsigned bits[16];
#pragma unroll
        for (int e = 0; e < 16; ++e) {
          const float av = a[e >> 2][e & 3];
          bal[e] = __builtin_amdgcn_ballot_w64(!(av <= 0.f));
          bits[e] = __builtin_bit_cast(unsigned, av) & 0x7fffffffu;
        }
        asm volatile("s_nop 1\n\tv_cndmask_b32 %0, %1, %0, %2" : "+v"(bits[15]) : "v"(0x80000000u), "s"(bal[15]));
        bal[15] |= 0xaaaaaaaaaaaaaaaaull;
        unsigned mask = 0;
        asm volatile("s_nop 1" ::: "memory");
#pragma unroll
        for (int e = 0; e < 16; ++e)
          asm volatile("v_addc_co_u32 %0, vcc, %0, %0, %1" : "+v"(mask) : "s"(bal[e]) : "vcc");
        const unsigned cnt = __builtin_popcount(mask);
        const unsigned incl = wave_scan(cnt);
        const unsigned n = __builtin_amdgcn_readlane(incl, 63);
        unsigned ad = (unsigned)(size_t)(LDS_AS char*)(buf + 128) + (incl - cnt) * 4u;
#pragma unroll
        for (int q = 0; q < 4; ++q)
          asm volatile("s_mov_b64 exec, %5\n\tds_write_b32 %0, %1\n\tv_add_u32 %0, 4, %0\n\t"
                       "s_mov_b64 exec, %6\n\tds_write_b32 %0, %2\n\tv_add_u32 %0, 4, %0\n\t"
                       "s_mov_b64 exec, %7\n\tds_write_b32 %0, %3\n\tv_add_u32 %0, 4, %0\n\t"
                       "s_mov_b64 exec, %8\n\tds_write_b32 %0, %4\n\tv_add_u32 %0, 4, %0\n\t"
                       "s_mov_b64 exec, -1"
                       : "+v"(ad) : "v"(bits[4 * q]), "v"(bits[4 * q + 1]), "v"(bits[4 * q + 2]), "v"(bits[4 * q + 3]),
                         "s"(bal[4 * q]), "s"(bal[4 * q + 1]), "s"(bal[4 * q + 2]), "s"(bal[4 * q + 3]) : "memory");
        *(unsigned short*)(buf + (xvo >> 3)) = (unsigned short)mask;
        const __amdgpu_buffer_rsrc_t rd = __builtin_amdgcn_make_buffer_rsrc((void*)(dstp + j * MSG_BYTES), 0, (int)((32u + n) * 4u), 0x00020000);
        asm volatile("s_waitcnt lgkmcnt(0)" ::: "memory");
        u32x4 v[NST];
#pragma unroll
        for (int s = 0; s < NST; ++s) v[s] = *(u32x4*)(buf + s * 1024 + xvo);
#pragma unroll
        for (int s = 0; s < NST; ++s) __builtin_amdgcn_raw_buffer_store_b128(v[s], rd, xvo + s * 1024, 0, 17);
      }
#pragma unroll
      for (int t = 0; t < 4; ++t) __builtin_amdgcn_raw_buffer_store_b128(u32x4{0u, 0u, 0u, 0u}, rxch, xvo + xo + t * 1024, 0, 16);
      SB();
    };

    f32x4 held[4];
#pragma unroll
    for (int t = 0; t < 4; ++t) held[t] = f32x4{0.f, 0.f, 0.f, 0.f};
    for (int it = 0; it < 8; ++it) {
      const int jA = 2 * it + (kap ^ 1), jP = 2 * it - 2 + kap;
      u32x4 pp[4];
      const unsigned po = it > 0 ? (unsigned)jP * 4096u : OOB - 4096u;
#pragma unroll
      for (int t = 0; t < 4; ++t) pp[t] = __builtin_amdgcn_raw_buffer_load_b128(rxch, xvo + t * 1024, po, 16);
      asm volatile("" ::: "memory");
      f32x4 acc[4];
      gather(2 * it, acc);
#pragma unroll
      for (int t = 0; t < 4; ++t) {
        u32x4 v = __builtin_bit_cast(u32x4, acc[t]);
#pragma unroll
        for (int rg = 0; rg < 4; ++rg) v[rg] |= (v[rg] - 1u) & ~v[rg] & 0x80000000u;
        __builtin_amdgcn_raw_buffer_store_b128(v, rxch, xvo + (unsigned)jA * 4096u + t * 1024, 0, 16);
      }
      SB();
      asm volatile("" : "+v"(pp[0]), "+v"(pp[1]), "+v"(pp[2]), "+v"(pp[3]));
      if (it > 0) finalize(jP, held, pp);
      if (it == 1) STAMP(3);
      gather(2 * it + 1, held);
    }
    STAMP(4);
    {
      u32x4 pp[4];
#pragma unroll
      for (int t = 0; t < 4; ++t) pp[t] = u32x4{0u, 0u, 0u, 0u};
      finalize(14 + kap, held, pp);
    }
    if (lane == 0) __hip_atomic_store((unsigned*)seqp, call, __ATOMIC_RELAXED, __HIP_MEMORY_SCOPE_AGENT);
    STAMP(5);
  }
  if (dd != 0 && wv < 3) {
    char* stage = smem + LDS_STAGE + wv * 5120;
    const char* mbase = myws + WS_MSG + (size_t)(((call & 1) * 8 + src) * 16 + tile) * MSG_TILE;
    const __amdgpu_buffer_rsrc_t rm = __builtin_amdgcn_make_buffer_rsrc((void*)mbase, 0, MSG_TILE, 0x00020000);
    const __amdgpu_buffer_rsrc_t ro = __builtin_amdgcn_make_buffer_rsrc((void*)(P.out + (size_t)(src * M_SH + rho * BM) * N_SH), 0, BM * N_SH * 4, 0x00020000);
    for (int m = wv; m < 8; m += 3) {
      const int j = 2 * m + kap;
      const unsigned mo = (unsigned)(j * MSG_BYTES);
      unsigned val[16];
      unsigned n;
      for (;;) {
        u32x4 mm[NST];
#pragma unroll
        for (int s = 0; s < NST; ++s) {
          const unsigned o = s * 1024 + lane * 16;
          mm[s] = __builtin_amdgcn_raw_buffer_load_b128(rm, o < MSG_BYTES ? mo + o : OOB, 0, 17);
        }
#pragma unroll
        for (int s = 0; s < NST; ++s) *(u32x4*)(stage + s * 1024 + lane * 16) = mm[s];
        asm volatile("s_waitcnt lgkmcnt(0)" ::: "memory");
        const unsigned mask = ((volatile unsigned short*)stage)[lane];
        const unsigned cnt = __builtin_popcount(mask);
        const unsigned incl = wave_scan(cnt);
        n = __builtin_amdgcn_readlane(incl, 63);
        unsigned ad = (unsigned)(size_t)(LDS_AS char*)(stage + 128) + (incl - cnt) * 4u;
        u64 bal[16];
        unsigned ms = mask << 16;
#pragma unroll
        for (int e = 0; e < 16; ++e) { asm volatile("v_add_co_u32 %0, %1, %0, %0" : "+v"(ms), "=s"(bal[e])); val[e] = 0x80000000u; }
#pragma unroll
        for (int q = 0; q < 4; ++q)
          asm volatile("s_mov_b64 exec, %5\n\tds_read_b32 %1, %0\n\tv_add_u32 %0, 4, %0\n\t"
                       "s_mov_b64 exec, %6\n\tds_read_b32 %2, %0\n\tv_add_u32 %0, 4, %0\n\t"
                       "s_mov_b64 exec, %7\n\tds_read_b32 %3, %0\n\tv_add_u32 %0, 4, %0\n\t"
                       "s_mov_b64 exec, %8\n\tds_read_b32 %4, %0\n\tv_add_u32 %0, 4, %0\n\t"
                       "s_mov_b64 exec, -1\n\ts_waitcnt lgkmcnt(0)"
                       : "+v"(ad), "+v"(val[4 * q]), "+v"(val[4 * q + 1]), "+v"(val[4 * q + 2]), "+v"(val[4 * q + 3])
                       : "s"(bal[4 * q]), "s"(bal[4 * q + 1]), "s"(bal[4 * q + 2]), "s"(bal[4 * q + 3]) : "memory");
        unsigned mn = (lane & 1) ? (mask & 1u) : 1u;
#pragma unroll
        for (int e = 0; e < 16; ++e) mn = min(mn, val[e]);
        if (__all(mn != 0u)) break;
        if (TIMED_OUT()) { bad = true; break; }
        __builtin_amdgcn_s_sleep(1);
      }
      const unsigned ovo = (unsigned)((j * 16 + 4 * g) * N_SH + strip * BN + 4 * c) * 4u;
#pragma unroll
      for (int rg = 0; rg < 4; ++rg) {
        u32x4 v;
#pragma unroll
        for (int t = 0; t < 4; ++t) v[t] = (unsigned)max((int)val[t * 4 + rg], 0);
        if (bad) v[0] = 0x7fc00000u;
        __builtin_amdgcn_raw_buffer_store_b128(v, ro, ovo + rg * (N_SH * 4), 0, 0);
      }
      const unsigned nb = (32u + n) * 4u;
#pragma unroll
      for (int s = 0; s < NST; ++s) {
        const unsigned o = s * 1024 + lane * 16;
        __builtin_amdgcn_raw_buffer_store_b128(u32x4{0u, 0u, 0u, 0u}, rm, o < nb ? mo + o : OOB, 0, 17);
      }
    }
  }
  STAMP(7);
  STAMP_FLUSH();
}

extern "C" void kernel_launch(int rank, int nranks, ncclComm_t comm, hipStream_t stream,
                              void* const* d_in, const int* in_sizes, int n_in,
                              void* d_out, int out_size, const int* in_dtypes, int out_dtype,
                              void* const* d_out_all, void* const* d_ws_all, size_t ws_bytes) {
  if (nranks != NRANK || (unsigned)rank >= (unsigned)NRANK || ws_bytes < WS_NEED) return;
  Params P;
  P.x = (const float*)d_in[0]; P.w = (const float*)d_in[1]; P.rank = rank; P.out = (float*)d_out;
  for (int p = 0; p < NRANK; ++p) P.ws[p] = (char*)d_ws_all[p];
  P.myws = P.ws[rank];
  HIP_CHECK(hipFuncSetAttribute((const void*)gemm_a2a, hipFuncAttributeMaxDynamicSharedMemorySize, LDS_BYTES));
  gemm_a2a<<<NWG, 256, LDS_BYTES, stream>>>(P);
  HIP_CHECK(hipGetLastError());
}
```

```cpp
#include <hip/hip_runtime.h>
#include <rccl/rccl.h>
#include <cstdio>

#define HIP_CHECK(expr) do { hipError_t _e = (expr); if (_e != hipSuccess) { \
    fprintf(stderr, "HIP error %s at %s:%d\n", hipGetErrorString(_e), __FILE__, __LINE__); \
    return; } } while (0)

typedef float f32x4 __attribute__((ext_vector_type(4)));
typedef _Float16 f16x2 __attribute__((ext_vector_type(2)));
typedef unsigned u32x4 __attribute__((ext_vector_type(4)));
typedef unsigned long long u64;
#define LDS_AS __attribute__((address_space(3)))

constexpr int NRANK = 8;
constexpr int M_SH = 512, K_DIM = 4096, N_DIM = 4096, N_SH = 512;
constexpr int BM = 256, BN = 64;
constexpr int NWG = 256;
#define EARLY 1
#define XW KC
#define PACE 135
#define PACEWIN 300000ull
#define XCHAUX 16
#define OAUX 16
#define MLAUX 17
#define SWR 8
#define KC 20
constexpr int KSV = 64 - 3 * KC;
constexpr int NST = 5;
constexpr u64 DEADLINE_TICKS = 50000000ull;
constexpr int WS_XFLAG = 4096;
constexpr int WS_SEQ = 49152;
constexpr int WS_DBG = 65536;
constexpr int WS_X16 = 1 << 20;
constexpr int WS_XCH = 10 << 20;
constexpr int MSG_BYTES = 4224;
constexpr int WS_MSG = 18 << 20;
constexpr int MSG_TILE = 16 * MSG_BYTES;
constexpr size_t WS_NEED = (size_t)WS_MSG + 2ull * 8 * 16 * MSG_TILE;
constexpr int RED_BYTES = 3 * 4096;
constexpr int NRING = 6;
constexpr int LDS_OWN = NRING * RED_BYTES;
constexpr int LDS_STAGE = LDS_OWN + 16 * 4096;
constexpr int LDS_CNT = LDS_STAGE + 4 * 5120;
constexpr int LDS_BYTES = LDS_CNT + 256;
constexpr unsigned OOB = 0x7ffffff0u;

struct Params {
  const float* x; const float* w;
  float* out;
  char* myws;
  char* ws[NRANK];
  int rank;
};

__device__ __forceinline__ unsigned pk(float a, float b) {
  f16x2 r; r[0] = (_Float16)a; r[1] = (_Float16)b; return __builtin_bit_cast(unsigned, r);
}
__device__ __forceinline__ u32x4 make_srd(const void* p, unsigned bytes) {
  const u64 a = (u64)p;
  u32x4 r;
  r[0] = __builtin_amdgcn_readfirstlane((unsigned)a);
  r[1] = __builtin_amdgcn_readfirstlane((unsigned)(a >> 32)) & 0xffffu;
  r[2] = bytes; r[3] = 0x00020000u;
  return r;
}
__device__ __forceinline__ unsigned wave_scan(unsigned v) {
  v += __builtin_amdgcn_update_dpp(0u, v, 0x111, 0xf, 0xf, false);
  v += __builtin_amdgcn_update_dpp(0u, v, 0x112, 0xf, 0xf, false);
  v += __builtin_amdgcn_update_dpp(0u, v, 0x114, 0xf, 0xf, false);
  v += __builtin_amdgcn_update_dpp(0u, v, 0x118, 0xf, 0xf, false);
  v += __builtin_amdgcn_update_dpp(0u, v, 0x142, 0xa, 0xf, false);
  v += __builtin_amdgcn_update_dpp(0u, v, 0x143, 0xc, 0xf, false);
  return v;
}
#define STAMP(i)
#define STAMP_FLUSH()
#define STAMP_DECL
#define SB() __builtin_amdgcn_sched_barrier(0)
#define HMARK(n)
#define TIMED_OUT() (__builtin_amdgcn_s_memrealtime() - t_start > DEADLINE_TICKS)

__global__ void __launch_bounds__(256, 1) gemm_a2a(Params P) {
  extern __shared__ __attribute__((aligned(16))) char smem[];
  const int tid = threadIdx.x, lane = tid & 63, wv = __builtin_amdgcn_readfirstlane(tid >> 6);
  const int bid = blockIdx.x;
  const int idx = bid >> 3;
  const int kap = bid & 1, rho = idx & 1, dd = idx >> 2;
  const int strip = (bid & 6) + ((idx >> 1) & 1);
  const int dest = (P.rank + dd) & 7, src = (P.rank - dd) & 7;
  const int tile = rho * 8 + strip, pid = dd * 16 + tile;
  char* myws = P.myws;
  const int c = lane & 15, g = lane >> 4;
  STAMP_DECL
  STAMP(0);
  const u64 t_start = __builtin_amdgcn_s_memrealtime();
  volatile LDS_AS unsigned* cnts = (volatile LDS_AS unsigned*)(smem + LDS_CNT);
  const char* seqp = myws + WS_SEQ + 64 * bid;
  unsigned call;
  u32x4 seq;
  if (tid < 32) ((unsigned*)(smem + LDS_CNT))[tid] = 0u;

  const u32x4 rw = make_srd(P.w, K_DIM * N_DIM * 4);
  u32x4 rxs;
#define BLOAD(dst, vo, rs, so, imm, aux) asm volatile("buffer_load_dwordx4 %0, %1, %2, %3 offen offset:%4" aux : "=v"(dst) : "v"(vo), "s"(rs), "s"(so), "n"(imm) : "memory")
#define BSTORE(v, vo, rs, so, imm, aux) asm volatile("buffer_store_dwordx4 %0, %1, %2, %3 offen offset:%4" aux "\n\ts_nop 1" :: "v"(v), "v"(vo), "s"(rs), "s"(so), "n"(imm) : "memory")
#define WAIT1(n, a) asm volatile("s_waitcnt vmcnt(%1)" : "+v"(a) : "n"(n) : "memory")
#define WAIT2(n, a, b) asm volatile("s_waitcnt vmcnt(%2)" : "+v"(a), "+v"(b) : "n"(n) : "memory")
  const unsigned wvo = (unsigned)(4 * g * N_DIM + dest * N_SH + strip * BN + 4 * c) * 4u;
  const unsigned wso = (unsigned)(kap * 2048 + (wv < 3 ? wv : 3 * KC) * 32) * (N_DIM * 4u);
  const unsigned wsoA = wso + rho * (16 * N_DIM * 4u), wsoB = wso + (1 - rho) * (16 * N_DIM * 4u);
  const unsigned xvo = lane * 16;
  bool bad = false;
  unsigned* const xflag = (unsigned*)(myws + WS_XFLAG + 8192 * (rho * 2 + kap) + 128 * lane);
#define WAUX ""
#define WLOAD(i) do { if ((i) < 8) wl[i] = w0[(i) & 7]; else WLOAD_(i); } while (0)
#define WLOAD_(i) BLOAD(wl[i], wvo, rw, (((i) & 4) ? wsoB : wsoA) + (unsigned)(((i) >> 3) * WKS * 32 + ((i) & 3)) * (N_DIM * 4u), 0, WAUX)
#define XLOAD(s, q) BLOAD(win[s], xvo, rxs, (unsigned)(((q) >> 2) * 4096), ((q) & 3) * 1024, " sc1")
#define WCVT(i, jj) do { _Pragma("unroll") for (int t = 0; t < 4; ++t) { bb[t][jj] = pk(wl[i][t], wl[(i) + 1][t]); asm volatile("" : "+v"(bb[t][jj])); } SB(); } while (0)
#define WKEEP_(C, ks) do { _Pragma("unroll") for (int t = 0; t < 4; ++t) { u32x4 b_ = {bb[t][0], bb[t][1], bb[t][2], bb[t][3]}; \
    asm volatile("" : C(b_)); breg[ks][t] = b_; } SB(); } while (0)
#define WKEEPA(ks) WKEEP_("+a", ks)
#define WKEEPV(ks) WKEEP_("+v", ks)
#define SEQREQ() do { asm volatile("s_waitcnt lgkmcnt(0)" ::: "memory"); __builtin_amdgcn_s_barrier();                \
    asm volatile("s_load_dwordx4 %0, %1, 0x0" : "=s"(seq) : "s"(seqp) : "memory"); } while (0)
#define GETCALL() do { asm volatile("s_waitcnt lgkmcnt(0)" : "+s"(seq) :: "memory"); call = seq[0] + 1u;              \
    rxs = make_srd(myws + WS_X16 + (size_t)(call & 1) * (M_SH * K_DIM * 2) + (rho * 2 + kap) * 1048576 + wv * (16 * KC * 1024), \
                   (wv < 3 ? 16 * KC : 16 * KSV) * 1024u);                                                           \
    asm volatile("s_nop 4" ::: "memory"); } while (0)
#define XPOLL() do { while (!__all(__hip_atomic_load(xflag, __ATOMIC_RELAXED, __HIP_MEMORY_SCOPE_AGENT) >= call)) {   \
      if (TIMED_OUT()) { bad = true; break; }                                                                        \
      __builtin_amdgcn_s_sleep(4); } } while (0)
#define MF(i, CIN) "v_mfma_f32_16x16x32_f16 %[c" #i "], %[a], %[b" #i "], " CIN(i) "\n\t"
#define CIN_ACC(i) "%[c" #i "]"
#define CIN_ZERO(i) "0"
#define COUT(CC, A) [c0] CC(A[0]), [c1] CC(A[1]), [c2] CC(A[2]), [c3] CC(A[3])
#define BC_A "a"
#define BC_V "v"
#define BIN(K, ks) [b0] BC_##K(breg[ks][0]), [b1] BC_##K(breg[ks][1]), [b2] BC_##K(breg[ks][2]), [b3] BC_##K(breg[ks][3])
#define STEP_(K, CIN, CC, A, n_, s, sp, sn, so_, imm_, ks) do { asm volatile(MF(0, CIN)                               \
    "buffer_load_dwordx4 %[p], %[vo], %[rs], %[so] offen offset:%[imm] sc1\n\t" MF(1, CIN) MF(2, CIN)                \
    "s_waitcnt vmcnt(%[n])\n\t" MF(3, CIN)                                                                           \
    : COUT(CC, A), [p] "+v"(win[sp]), [nx] "+v"(win[sn])                                                             \
    : [a] "v"(win[s]), BIN(K, ks), [vo] "v"(xvo), [rs] "s"(rxs), [so] "s"(so_), [imm] "n"(imm_), [n] "n"(n_) : "memory"); SB(); } while (0)
#define MFMA_FENCE(A) asm volatile("s_nop 7\n\ts_nop 1" : "+v"(A[0]), "+v"(A[1]), "+v"(A[2]), "+v"(A[3]))

  f32x4 w0[8];
  asm volatile("s_nop 4" ::: "memory");
#pragma unroll
  for (int i = 0; i < 8; ++i) BLOAD(w0[i], wvo, rw, ((i & 4) ? wsoB : wsoA) + (unsigned)(i & 3) * (N_DIM * 4u), 0, WAUX);
  SB();
  if (wv < 3) {
    constexpr int WKS = 3;
    f32x4 wl[8 * KC];
    u32x4 win[XW];
    u32x4 breg[KC][4];
    f32x4 eacc[EARLY][4];
    unsigned xc = 0;
#define P1_INIT() SEQREQ()
#define P1_CLOAD() do { GETCALL(); asm volatile("global_load_dword %0, %1, off sc1" : "=v"(xc) : "v"(xflag) : "memory"); SB(); } while (0)
#define P1_CHECK() do { asm volatile("" : "+v"(xc)); if (!__all(xc >= call)) XPOLL(); asm volatile("" ::: "memory");  \
    if (tid == 0) cnts[17] = 1u; SB(); STAMP(1); } while (0)
#define EMFMA_(K, CIN, CC, n_, s, e, ks) do { asm volatile("s_waitcnt vmcnt(%[n])\n\ts_nop 0\n\t" MF(0, CIN) MF(1, CIN) MF(2, CIN) MF(3, CIN) \
    : COUT(CC, eacc[e]), [a] "+v"(win[s]) : BIN(K, ks), [n] "n"(n_) : "memory"); SB(); } while (0)
#define EMFMA0(K, n_, s, e, ks) EMFMA_(K, CIN_ZERO, "=&v", n_, s, e, ks)
#define EMFMA(K, n_, s, e, ks) EMFMA_(K, CIN_ACC, "+v", n_, s, e, ks)
#define ESTEP(K, n_, s, sp, sn, q, e, ks) STEP_(K, CIN_ACC, "+v", eacc[e], n_, s, sp, sn, (unsigned)(((q) >> 2) * 4096), ((q) & 3) * 1024, ks)
    asm volatile("s_nop 4" ::: "memory");
    WLOAD(0);
    WLOAD(1);
    WLOAD(2);
    WLOAD(3);
    WLOAD(4);
    WLOAD(5);
    WLOAD(6);
    WLOAD(7);
    WLOAD(8);
    WLOAD(9);
    SB();
    P1_INIT();
    { unsigned bb[4][4];
    WAIT2(8, wl[0], wl[1]);
    WCVT(0, 0);
    WLOAD(10);
    WLOAD(11);
    SB();
    WAIT2(8, wl[2], wl[3]);
    WCVT(2, 1);
    WLOAD(12);
    WLOAD(13);
    SB();
    WAIT2(8, wl[4], wl[5]);
    WCVT(4, 2);
    WLOAD(14);
    WLOAD(15);
    SB();
    WAIT2(8, wl[6], wl[7]);
    WCVT(6, 3);
    WLOAD(16);
    WLOAD(17);
    SB();
    WKEEPA(0); }
    { unsigned bb[4][4];
    WAIT2(8, wl[8], wl[9]);
    WCVT(8, 0);
    WLOAD(18);
    WLOAD(19);
    SB();
    WAIT2(8, wl[10], wl[11]);
    WCVT(10, 1);
    WLOAD(20);
    WLOAD(21);
    SB();
    WAIT2(8, wl[12], wl[13]);
    WCVT(12, 2);
    WLOAD(22);
    WLOAD(23);
    SB();
    WAIT2(8, wl[14], wl[15]);
    WCVT(14, 3);
    WLOAD(24);
    WLOAD(25);
    SB();
    WKEEPA(1); }
    { unsigned bb[4][4];
    WAIT2(8, wl[16], wl[17]);
    WCVT(16, 0);
    WLOAD(26);
    WLOAD(27);
    SB();
    WAIT2(8, wl[18], wl[19]);
    WCVT(18, 1);
    WLOAD(28);
    WLOAD(29);
    SB();
    WAIT2(8, wl[20], wl[21]);
    WCVT(20, 2);
    WLOAD(30);
    WLOAD(31);
    SB();
    WAIT2(8, wl[22], wl[23]);
    WCVT(22, 3);
    WLOAD(32);
    WLOAD(33);
    SB();
    WKEEPA(2); }
    { unsigned bb[4][4];
    WAIT2(8, wl[24], wl[25]);
    WCVT(24, 0);
    WLOAD(34);
    WLOAD(35);
    SB();
    WAIT2(8, wl[26], wl[27]);
    WCVT(26, 1);
    WLOAD(36);
    WLOAD(37);
    SB();
    WAIT2(8, wl[28], wl[29]);
    WCVT(28, 2);
    WLOAD(38);
    WLOAD(39);
    SB();
    WAIT2(8, wl[30], wl[31]);
    WCVT(30, 3);
    WLOAD(40);
    WLOAD(41);
    SB();
    WKEEPA(3); }
    { unsigned bb[4][4];
    WAIT2(8, wl[32], wl[33]);
    WCVT(32, 0);
    WLOAD(42);
    WLOAD(43);
    SB();
    WAIT2(8, wl[34], wl[35]);
    WCVT(34, 1);
    WLOAD(44);
    WLOAD(45);
    SB();
    WAIT2(8, wl[36], wl[37]);
    WCVT(36, 2);
    WLOAD(46);
    WLOAD(47);
    SB();
    WAIT2(8, wl[38], wl[39]);
    WCVT(38, 3);
    WLOAD(48);
    WLOAD(49);
    SB();
    WKEEPA(4); }
    { unsigned bb[4][4];
    WAIT2(8, wl[40], wl[41]);
    WCVT(40, 0);
    WLOAD(50);
    WLOAD(51);
    SB();
    WAIT2(8, wl[42], wl[43]);
    WCVT(42, 1);
    WLOAD(52);
    WLOAD(53);
    SB();
    WAIT2(8, wl[44], wl[45]);
    WCVT(44, 2);
    WLOAD(54);
    WLOAD(55);
    SB();
    WAIT2(8, wl[46], wl[47]);
    WCVT(46, 3);
    WLOAD(56);
    WLOAD(57);
    SB();
    WKEEPA(5); }
    { unsigned bb[4][4];
    WAIT2(8, wl[48], wl[49]);
    WCVT(48, 0);
    WLOAD(58);
    WLOAD(59);
    SB();
    WAIT2(8, wl[50], wl[51]);
    WCVT(50, 1);
    WLOAD(60);
    WLOAD(61);
    SB();
    WAIT2(8, wl[52], wl[53]);
    WCVT(52, 2);
    WLOAD(62);
    WLOAD(63);
    SB();
    WAIT2(8, wl[54], wl[55]);
    WCVT(54, 3);
    WLOAD(64);
    WLOAD(65);
    SB();
    WKEEPA(6); }
    { unsigned bb[4][4];
    WAIT2(8, wl[56], wl[57]);
    WCVT(56, 0);
    WLOAD(66);
    WLOAD(67);
    SB();
    WAIT2(8, wl[58], wl[59]);
    WCVT(58, 1);
    WLOAD(68);
    WLOAD(69);
    SB();
    WAIT2(8, wl[60], wl[61]);
    WCVT(60, 2);
    WLOAD(70);
    WLOAD(71);
    SB();
    WAIT2(8, wl[62], wl[63]);
    WCVT(62, 3);
    WLOAD(72);
    WLOAD(73);
    SB();
    WKEEPA(7); }
    { unsigned bb[4][4];
    WAIT2(8, wl[64], wl[65]);
    WCVT(64, 0);
    WLOAD(74);
    WLOAD(75);
    SB();
    WAIT2(8, wl[66], wl[67]);
    WCVT(66, 1);
    WLOAD(76);
    WLOAD(77);
    SB();
    WAIT2(8, wl[68], wl[69]);
    WCVT(68, 2);
    WLOAD(78);
    WLOAD(79);
    SB();
    WAIT2(8, wl[70], wl[71]);
    WCVT(70, 3);
    WLOAD(80);
    WLOAD(81);
    SB();
    WKEEPA(8); }
    { unsigned bb[4][4];
    WAIT2(8, wl[72], wl[73]);
    WCVT(72, 0);
    WLOAD(82);
    WLOAD(83);
    SB();
    WAIT2(8, wl[74], wl[75]);
    WCVT(74, 1);
    WLOAD(84);
    WLOAD(85);
    SB();
    WAIT2(8, wl[76], wl[77]);
    WCVT(76, 2);
    WLOAD(86);
    WLOAD(87);
    SB();
    WAIT2(8, wl[78], wl[79]);
    WCVT(78, 3);
    WLOAD(88);
    WLOAD(89);
    SB();
    WKEEPA(9); }
    { unsigned bb[4][4];
    WAIT2(8, wl[80], wl[81]);
    WCVT(80, 0);
    WLOAD(90);
    WLOAD(91);
    SB();
    WAIT2(8, wl[82], wl[83]);
    WCVT(82, 1);
    WLOAD(92);
    WLOAD(93);
    SB();
    WAIT2(8, wl[84], wl[85]);
    WCVT(84, 2);
    WLOAD(94);
    WLOAD(95);
    SB();
    WAIT2(8, wl[86], wl[87]);
    WCVT(86, 3);
    WLOAD(96);
    WLOAD(97);
    SB();
    WKEEPA(10); }
    { unsigned bb[4][4];
    WAIT2(8, wl[88], wl[89]);
    WCVT(88, 0);
    WLOAD(98);
    WLOAD(99);
    SB();
    WAIT2(8, wl[90], wl[91]);
    WCVT(90, 1);
    WLOAD(100);
    WLOAD(101);
    SB();
    WAIT2(8, wl[92], wl[93]);
    WCVT(92, 2);
    WLOAD(102);
    WLOAD(103);
    SB();
    WAIT2(8, wl[94], wl[95]);
    WCVT(94, 3);
    WLOAD(104);
    WLOAD(105);
    SB();
    WKEEPA(11); }
    { unsigned bb[4][4];
    WAIT2(8, wl[96], wl[97]);
    WCVT(96, 0);
    WLOAD(106);
    WLOAD(107);
    SB();
    WAIT2(8, wl[98], wl[99]);
    WCVT(98, 1);
    WLOAD(108);
    WLOAD(109);
    SB();
    WAIT2(8, wl[100], wl[101]);
    WCVT(100, 2);
    WLOAD(110);
    WLOAD(111);
    SB();
    WAIT2(8, wl[102], wl[103]);
    WCVT(102, 3);
    WLOAD(112);
    WLOAD(113);
    SB();
    WKEEPA(12); }
    { unsigned bb[4][4];
    WAIT2(8, wl[104], wl[105]);
    WCVT(104, 0);
    WLOAD(114);
    WLOAD(115);
    SB();
    WAIT2(8, wl[106], wl[107]);
    WCVT(106, 1);
    WLOAD(116);
    WLOAD(117);
    SB();
    WAIT2(8, wl[108], wl[109]);
    WCVT(108, 2);
    WLOAD(118);
    WLOAD(119);
    SB();
    WAIT2(8, wl[110], wl[111]);
    WCVT(110, 3);
    WLOAD(120);
    WLOAD(121);
    SB();
    WKEEPA(13); }
    { unsigned bb[4][4];
    WAIT2(8, wl[112], wl[113]);
    WCVT(112, 0);
    WLOAD(122);
    WLOAD(123);
    SB();
    WAIT2(8, wl[114], wl[115]);
    WCVT(114, 1);
    WLOAD(124);
    WLOAD(125);
    SB();
    WAIT2(8, wl[116], wl[117]);
    WCVT(116, 2);
    WLOAD(126);
    WLOAD(127);
    SB();
    WAIT2(8, wl[118], wl[119]);
    WCVT(118, 3);
    WLOAD(128);
    WLOAD(129);
    SB();
    WKEEPA(14); }
    { unsigned bb[4][4];
    WAIT2(8, wl[120], wl[121]);
    WCVT(120, 0);
    WLOAD(130);
    WLOAD(131);
    SB();
    WAIT2(8, wl[122], wl[123]);
    WCVT(122, 1);
    WLOAD(132);
    WLOAD(133);
    SB();
    WAIT2(8, wl[124], wl[125]);
    WCVT(124, 2);
    WLOAD(134);
    WLOAD(135);
    SB();
    WAIT2(8, wl[126], wl[127]);
    WCVT(126, 3);
    WLOAD(136);
    WLOAD(137);
    SB();
    WKEEPA(15); }
    { unsigned bb[4][4];
    WAIT2(8, wl[128], wl[129]);
    WCVT(128, 0);
    WLOAD(138);
    WLOAD(139);
    SB();
    P1_CLOAD();
    WAIT2(9, wl[130], wl[131]);
    WCVT(130, 1);
    WLOAD(140);
    WLOAD(141);
    SB();
    WAIT2(9, wl[132], wl[133]);
    WCVT(132, 2);
    WLOAD(142);
    WLOAD(143);
    SB();
    WAIT2(9, wl[134], wl[135]);
    WCVT(134, 3);
    WLOAD(144);
    WLOAD(145);
    SB();
    WKEEPV(16); }
    { unsigned bb[4][4];
    WAIT2(9, wl[136], wl[137]);
    WCVT(136, 0);
    WLOAD(146);
    WLOAD(147);
    SB();
    WAIT2(9, wl[138], wl[139]);
    WCVT(138, 1);
    WLOAD(148);
    WLOAD(149);
    SB();
    WAIT2(8, wl[140], wl[141]);
    WCVT(140, 2);
    WLOAD(150);
    WLOAD(151);
    SB();
    P1_CHECK();
    XLOAD(0, 0);
    XLOAD(1, 1);
    WAIT2(10, wl[142], wl[143]);
    WCVT(142, 3);
    WLOAD(152);
    WLOAD(153);
    SB();
    WKEEPV(17); }
    XLOAD(2, 2);
    XLOAD(3, 3);
    { unsigned bb[4][4];
    WAIT2(12, wl[144], wl[145]);
    WCVT(144, 0);
    WLOAD(154);
    WLOAD(155);
    SB();
    XLOAD(4, 4);
    XLOAD(5, 5);
    WAIT2(14, wl[146], wl[147]);
    WCVT(146, 1);
    WLOAD(156);
    WLOAD(157);
    SB();
    XLOAD(6, 6);
    XLOAD(7, 7);
    WAIT2(16, wl[148], wl[149]);
    WCVT(148, 2);
    WLOAD(158);
    WLOAD(159);
    SB();
    XLOAD(8, 8);
    XLOAD(9, 9);
    WAIT2(18, wl[150], wl[151]);
    WCVT(150, 3);
    SB();
    WKEEPV(18); }
    EMFMA0(A, 17, 0, 0, 0);
    EMFMA(A, 16, 1, 0, 1);
    XLOAD(10, 10);
    XLOAD(11, 11);
    { unsigned bb[4][4];
    WAIT2(16, wl[152], wl[153]);
    WCVT(152, 0);
    SB();
    EMFMA(A, 15, 2, 0, 2);
    EMFMA(A, 14, 3, 0, 3);
    XLOAD(12, 12);
    XLOAD(13, 13);
    WAIT2(14, wl[154], wl[155]);
    WCVT(154, 1);
    SB();
    EMFMA(A, 13, 4, 0, 4);
    EMFMA(A, 12, 5, 0, 5);
    XLOAD(14, 14);
    XLOAD(15, 15);
    WAIT2(12, wl[156], wl[157]);
    WCVT(156, 2);
    SB();
    EMFMA(A, 11, 6, 0, 6);
    EMFMA(A, 10, 7, 0, 7);
    XLOAD(16, 16);
    XLOAD(17, 17);
    WAIT2(10, wl[158], wl[159]);
    WCVT(158, 3);
    SB();
    WKEEPV(19); }
    EMFMA(A, 9, 8, 0, 8);
    EMFMA(A, 8, 9, 0, 9);
    XLOAD(18, 18);
    XLOAD(19, 19);
    STAMP(3);
    EMFMA(A, 9, 10, 0, 10);
    XLOAD(0, 20);
    XLOAD(1, 21);
    EMFMA(A, 10, 11, 0, 11);
    XLOAD(2, 22);
    XLOAD(3, 23);
    EMFMA(A, 11, 12, 0, 12);
    XLOAD(4, 24);
    XLOAD(5, 25);
    EMFMA(A, 12, 13, 0, 13);
    XLOAD(6, 26);
    XLOAD(7, 27);
    EMFMA(A, 13, 14, 0, 14);
    XLOAD(8, 28);
    XLOAD(9, 29);
    EMFMA(A, 14, 15, 0, 15);
    XLOAD(10, 30);
    XLOAD(11, 31);
    EMFMA(V, 15, 16, 0, 16);
    XLOAD(12, 32);
    XLOAD(13, 33);
    EMFMA(V, 16, 17, 0, 17);
    XLOAD(14, 34);
    XLOAD(15, 35);
    EMFMA(V, 17, 18, 0, 18);
    XLOAD(16, 36);
    XLOAD(17, 37);
    WAIT1(18, win[19]);
    asm volatile("s_nop 1" ::: "memory");
    ESTEP(V, 18, 19, 18, 0, 38, 0, 19);
    STAMP(2);
    HMARK(4);
    static_assert(EARLY == 1, "chunk 0 comes out of phase 1");
    const bool paced = t_start - (((u64)seq[3] << 32) | seq[2]) < PACEWIN;
    u64 tchunk = __builtin_amdgcn_s_memrealtime();
    const unsigned cnt_ad = (unsigned)(size_t)(LDS_AS char*)(smem + LDS_CNT);
    f32x4 accB[4];
#define PUT_CHECK(o_) do { if ((int)__builtin_amdgcn_readfirstlane(consumed) < (o_) - (NRING - 1)) {     \
        while ((int)cnts[16] < (o_) - (NRING - 1)) { if (TIMED_OUT()) { bad = true; break; } } }                      \
      asm volatile("" ::: "memory"); SB(); } while (0)
#define PUT_SIGNAL(o_) do { asm volatile("s_mov_b64 exec, 1\n\tds_add_u32 %0, %1\n\ts_mov_b64 exec, -1" :: "v"(cnt_ad + (unsigned)(o_) * 4u), "v"(1u) : "memory"); SB(); } while (0)
#define KSTEPS(AC, K0, K1)                                                                                           \
      _Pragma("unroll") for (int k = K0; k < K1; ++k) {                                                              \
        if (k == 0) STEP_(A, CIN_ZERO, "=&v", AC, XW - 2, k % XW, (k + XW - 1) % XW, (k + 1) % XW, qb + (k >> 2) * 4096, (k & 3) * 1024, k); \
        else if (k < 16) STEP_(A, CIN_ACC, "+v", AC, XW - 2, k % XW, (k + XW - 1) % XW, (k + 1) % XW, qb + (k >> 2) * 4096, (k & 3) * 1024, k); \
        else STEP_(V, CIN_ACC, "+v", AC, XW - 2, k % XW, (k + XW - 1) % XW, (k + 1) % XW, qb + (k >> 2) * 4096, (k & 3) * 1024, k); \
        if (k >= 2 && k < 6) { pw[(k - 2) * 64] = PVX[k - 2]; SB(); }                                                \
        if (k == 6) PUT_SIGNAL(o - 1);                                                                               \
        if (k == 12) { consumed = cnts[16]; SB(); }                                                                  \
      }
#define CHUNK(AC, PV, o_) do { const int o = (o_);                                                                   \
      const unsigned qb = (unsigned)(o * KC + XW - 1) * 1024u;           \
      f32x4* pw = (f32x4*)(smem + ((o - 1) % NRING) * RED_BYTES + wv * 4096 + xvo);                                  \
      f32x4 (&PVX)[4] = PV;                                                                                          \
      KSTEPS(AC, 0, 2)                                                                                               \
      PUT_CHECK(o - 1);                                                                                              \
      KSTEPS(AC, 2, KC)                                                                                              \
      if (paced) { tchunk += PACE; while (__builtin_amdgcn_s_memrealtime() < tchunk) __builtin_amdgcn_s_sleep(1); }  \
    } while (0)
    unsigned consumed = 0;
    for (int i = 0; i < 7; ++i) {
      CHUNK(accB, eacc[0], 2 * i + 1);
      CHUNK(eacc[0], accB, 2 * i + 2);
    }
    CHUNK(accB, eacc[0], 15);
    MFMA_FENCE(accB);
    PUT_CHECK(15);
    {
      f32x4* pw = (f32x4*)(smem + (15 % NRING) * RED_BYTES + wv * 4096 + xvo);
#pragma unroll
      for (int t = 0; t < 4; ++t) pw[t * 64] = accB[t];
      asm volatile("" ::: "memory");
      PUT_SIGNAL(15);
    }
    STAMP(4);
    HMARK(5);
    asm volatile("s_waitcnt vmcnt(0)" ::: "memory");
#pragma unroll
    for (int i = 0; i < XW; ++i) asm volatile("" :: "v"(win[i]));
  } else {
    constexpr int WKS = 1;
    constexpr int SW = 2 * KSV;
    f32x4 wl[8 * KSV];
    u32x4 win[SW];
    u32x4 breg[KSV][4];
    {
      const int rc = bid >> 3, kb = bid & 7;
      const u32x4 rx = make_srd(P.x, M_SH * K_DIM * 4);
      const unsigned xcvo = (unsigned)((rc * 16 + c) * K_DIM + kb * 512 + g * 4) * 4u;
      const unsigned xcvA = xcvo + (rc >> 4) * 64u, xcvB = xcvo + 64u - (rc >> 4) * 64u;
      f32x4 xl[32];
      asm volatile("s_nop 4" ::: "memory");
#pragma unroll
      for (int i = 0; i < 32; ++i) BLOAD(xl[i], (i & 1) ? xcvB : xcvA, rx, 0u, (i >> 1) * 128, "");
      SEQREQ();
      GETCALL();
      const int grp = (rc >> 4) * 2 + (kb >> 2), oc = (rc & 15) ^ ((kb >> 2) ^ 1);
      const u32x4 rx16w = make_srd(myws + WS_X16 + (size_t)(call & 1) * (M_SH * K_DIM * 2) + grp * 1048576, 1048576u);
      asm volatile("s_nop 4" ::: "memory");
#pragma unroll
      for (int i = 0; i < 16; ++i) {
        const int gk = (kb & 3) * 16 + i;
        const int u = gk >= 3 * KC ? 3 : gk % 3;
        const int ks = gk >= 3 * KC ? gk - 3 * KC : gk / 3;
        const int pos = u * 16 * KC + (u == 3 ? oc * KSV + ks : oc < EARLY ? ks * EARLY + oc : oc * KC + ks);
        WAIT2(30 - i, xl[2 * i], xl[2 * i + 1]);
        const f32x4 a_ = xl[2 * i], b_ = xl[2 * i + 1];
        u32x4 o_ = {pk(a_[0], a_[1]), pk(a_[2], a_[3]), pk(b_[0], b_[1]), pk(b_[2], b_[3])};
        BSTORE(o_, xvo, rx16w, (unsigned)pos * 1024u, 0, " sc1");
      }
      SB();
#pragma unroll
      for (int i = 0; i < 8 + SWR; ++i) WLOAD(i);
      asm volatile("s_waitcnt vmcnt(%0)" :: "n"(SWR) : "memory");
      if (lane == 0) {
        unsigned* f_ = (unsigned*)(myws + WS_XFLAG + 8192 * grp + 128 * ((rc & 15) * 4 + (kb & 3)));
        asm volatile("global_store_dword %0, %1, off sc1" :: "v"(f_), "v"(call) : "memory");
      }
      SB();
      STAMP(6);
    }
#pragma unroll
    for (int ks = 0; ks < KSV; ++ks) {
      unsigned bb[4][4];
#pragma unroll
      for (int jj = 0; jj < 4; ++jj) {
        const int i = ks * 8 + 2 * jj;
        WAIT2(i < 8 ? SWR : i + SWR <= 8 * KSV ? SWR - 2 : 8 * KSV - 2 - i, wl[i], wl[i + 1]);
        WCVT(i, jj);
        if (i >= 8 && i + SWR < 8 * KSV) { WLOAD(i + SWR); WLOAD(i + SWR + 1); }
        SB();
      }
      WKEEPA(ks);
    }
    STAMP(1);
    XPOLL();
    asm volatile("" ::: "memory");
#pragma unroll
    for (int i = 0; i < SW - 1; ++i) XLOAD(i, i);
    win[SW - 1] = u32x4{0u, 0u, 0u, 0u};
    WAIT1(SW - 2, win[0]);
    for (int it = 0; it < 8; ++it) {
      const unsigned qb = (unsigned)(it * SW + SW - 1) * 1024u;
#pragma unroll
      for (int h = 0; h < 2; ++h) {
        f32x4 acc[4];
#pragma unroll
        for (int k = h * KSV; k < (h + 1) * KSV; ++k) {
          if (k == h * KSV) STEP_(A, CIN_ZERO, "=&v", acc, SW - 2, k, (k + SW - 1) % SW, (k + 1) % SW, qb + (k >> 2) * 4096, (k & 3) * 1024, k - h * KSV);
          else STEP_(A, CIN_ACC, "+v", acc, SW - 2, k, (k + SW - 1) % SW, (k + 1) % SW, qb + (k >> 2) * 4096, (k & 3) * 1024, k - h * KSV);
        }
        MFMA_FENCE(acc);
        f32x4* pw = (f32x4*)(smem + LDS_OWN + (2 * it + h) * 4096 + xvo);
#pragma unroll
        for (int t = 0; t < 4; ++t) pw[t * 64] = acc[t];
        SB();
      }
    }
    asm volatile("s_waitcnt vmcnt(0)" ::: "memory");
#pragma unroll
    for (int i = 0; i < SW; ++i) asm volatile("" :: "v"(win[i]));
    STAMP(2);

    const bool local = dd == 0;
    char* dstp = local ? (char*)(P.out + (size_t)(P.rank * M_SH + rho * BM) * N_SH)
                       : P.ws[dest] + WS_MSG + (size_t)(((call & 1) * 8 + P.rank) * 16 + tile) * MSG_TILE;
    const __amdgpu_buffer_rsrc_t rxch = __builtin_amdgcn_make_buffer_rsrc((void*)(myws + WS_XCH + (size_t)pid * 65536), 0, 65536, 0x00020000);
    char* const buf = smem + LDS_STAGE + 3 * 5120;
    const unsigned tag = call & 1u;

    auto gather = [&](int o, f32x4 (&acc)[4]) {
      const f32x4* own = (const f32x4*)(smem + LDS_OWN + o * 4096 + xvo);
#pragma unroll
      for (int t = 0; t < 4; ++t) acc[t] = own[t * 64];
      while (cnts[o] < 3u) { if (TIMED_OUT()) { bad = true; break; } }
      asm volatile("" ::: "memory");
      const f32x4* part = (const f32x4*)(smem + (o % NRING) * RED_BYTES + xvo);
      f32x4 pr[12];
#pragma unroll
      for (int i = 0; i < 12; ++i) pr[i] = part[i * 64];
      SB();
#pragma unroll
      for (int t = 0; t < 4; ++t) {
        acc[t] = (acc[t] + pr[t]) + (pr[4 + t] + pr[8 + t]);
        asm volatile("" : "+v"(acc[t]) :: "memory");
      }
      if (lane == 0) cnts[16] = (unsigned)(o + 1);
      SB();
    };
    auto finalize = [&](int j, f32x4 (&a)[4], u32x4 (&pp)[4]) {
      const unsigned xo = (unsigned)j * 4096u;
      auto complete = [&]() {
        unsigned an = ~0u, orr = 0u;
#pragma unroll
        for (int t = 0; t < 4; ++t)
#pragma unroll
          for (int rg = 0; rg < 4; ++rg) { an &= pp[t][rg]; orr |= pp[t][rg]; }
        return __all(((tag ? an : ~orr) & 1u) != 0u);
      };
      if (!complete()) {
        do {
          if (TIMED_OUT()) { bad = true; break; }
#pragma unroll
          for (int t = 0; t < 4; ++t) pp[t] = __builtin_amdgcn_raw_buffer_load_b128(rxch, xvo + t * 1024, xo, XCHAUX);
          asm volatile("" ::: "memory");
        } while (!complete());
      }
#pragma unroll
      for (int t = 0; t < 4; ++t) a[t] += __builtin_bit_cast(f32x4, pp[t] & ~1u);
      if (bad) a[0][0] = __builtin_nanf("");
      if (local) {
        const __amdgpu_buffer_rsrc_t rd = __builtin_amdgcn_make_buffer_rsrc((void*)dstp, 0, BM * N_SH * 4, 0x00020000);
        const unsigned ovo = (xvo >> 8) * (4 * N_SH * 4) + (xvo & 255u);
        const unsigned so = (unsigned)(j * 16 * N_SH + strip * BN) * 4u;
#pragma unroll
        for (int rg = 0; rg < 4; ++rg) {
          f32x4 v;
#pragma unroll
          for (int t = 0; t < 4; ++t) v[t] = a[t][rg] <= 0.f ? 0.f : a[t][rg];
          __builtin_amdgcn_raw_buffer_store_b128(__builtin_bit_cast(u32x4, v), rd, ovo + so + rg * (N_SH * 4), 0, OAUX);
        }
      } else {
        u64 bal[16];
        unsigned bits[16];
#pragma unroll
        for (int e = 0; e < 16; ++e) {
          const float av = a[e >> 2][e & 3];
          bal[e] = __builtin_amdgcn_ballot_w64(!(av <= 0.f));
          bits[e] = __builtin_bit_cast(unsigned, av) & 0x7fffffffu;
        }
        asm volatile("s_nop 1\n\tv_cndmask_b32 %0, %1, %0, %2" : "+v"(bits[15]) : "v"(0x80000000u), "s"(bal[15]));
        bal[15] |= 0xaaaaaaaaaaaaaaaaull;
        unsigned mask = 0;
        asm volatile("s_nop 1" ::: "memory");
#pragma unroll
        for (int e = 0; e < 16; ++e)
          asm volatile("v_addc_co_u32 %0, vcc, %0, %0, %1" : "+v"(mask) : "s"(bal[e]) : "vcc");
        const unsigned cnt = __builtin_popcount(mask);
        const unsigned incl = wave_scan(cnt);
        const unsigned n = __builtin_amdgcn_readlane(incl, 63);
        unsigned ad = (unsigned)(size_t)(LDS_AS char*)(buf + 128) + (incl - cnt) * 4u;
#pragma unroll
        for (int q = 0; q < 4; ++q)
          asm volatile("s_mov_b64 exec, %5\n\tds_write_b32 %0, %1\n\tv_add_u32 %0, 4, %0\n\t"
                       "s_mov_b64 exec, %6\n\tds_write_b32 %0, %2\n\tv_add_u32 %0, 4, %0\n\t"
                       "s_mov_b64 exec, %7\n\tds_write_b32 %0, %3\n\tv_add_u32 %0, 4, %0\n\t"
                       "s_mov_b64 exec, %8\n\tds_write_b32 %0, %4\n\tv_add_u32 %0, 4, %0\n\t"
                       "s_mov_b64 exec, -1"
                       : "+v"(ad) : "v"(bits[4 * q]), "v"(bits[4 * q + 1]), "v"(bits[4 * q + 2]), "v"(bits[4 * q + 3]),
                         "s"(bal[4 * q]), "s"(bal[4 * q + 1]), "s"(bal[4 * q + 2]), "s"(bal[4 * q + 3]) : "memory");
        *(unsigned short*)(buf + (xvo >> 3)) = (unsigned short)mask;
        const __amdgpu_buffer_rsrc_t rd = __builtin_amdgcn_make_buffer_rsrc((void*)(dstp + j * MSG_BYTES), 0, (int)((32u + n) * 4u), 0x00020000);
        asm volatile("s_waitcnt lgkmcnt(0)" ::: "memory");
        u32x4 v[NST];
#pragma unroll
        for (int s = 0; s < NST; ++s) v[s] = *(u32x4*)(buf + s * 1024 + xvo);
#pragma unroll
        for (int s = 0; s < NST; ++s) __builtin_amdgcn_raw_buffer_store_b128(v[s], rd, xvo + s * 1024, 0, 17);
      }
      SB();
    };

    f32x4 held[4];
#pragma unroll
    for (int t = 0; t < 4; ++t) held[t] = f32x4{0.f, 0.f, 0.f, 0.f};
    for (int it = 0; it < 8; ++it) {
      const int jA = 2 * it + (kap ^ 1), jP = 2 * it - 2 + kap;
      u32x4 pp[4];
      const unsigned po = it > 0 ? (unsigned)jP * 4096u : OOB - 4096u;
#pragma unroll
      for (int t = 0; t < 4; ++t) pp[t] = __builtin_amdgcn_raw_buffer_load_b128(rxch, xvo + t * 1024, po, XCHAUX);
      asm volatile("" ::: "memory");
      f32x4 acc[4];
      gather(2 * it, acc);
#pragma unroll
      for (int t = 0; t < 4; ++t) {
        u32x4 v = __builtin_bit_cast(u32x4, acc[t]);
#pragma unroll
        for (int rg = 0; rg < 4; ++rg) v[rg] = (v[rg] & ~1u) | tag;
        __builtin_amdgcn_raw_buffer_store_b128(v, rxch, xvo + (unsigned)jA * 4096u + t * 1024, 0, 16);
      }
      SB();
      asm volatile("" : "+v"(pp[0]), "+v"(pp[1]), "+v"(pp[2]), "+v"(pp[3]));
      if (it > 0) finalize(jP, held, pp);
      if (it == 1) STAMP(3);
      if (it < 7) gather(2 * it + 1, held);
    }
    {
      u32x4 pp[4];
#pragma unroll
      for (int t = 0; t < 4; ++t) pp[t] = __builtin_amdgcn_raw_buffer_load_b128(rxch, xvo + t * 1024, (unsigned)(14 + kap) * 4096u, XCHAUX);
      asm volatile("" ::: "memory");
      gather(15, held);
      STAMP(4);
      finalize(14 + kap, held, pp);
    }
    if (lane == 0) {
      __hip_atomic_store((unsigned*)seqp, call, __ATOMIC_RELAXED, __HIP_MEMORY_SCOPE_AGENT);
      __hip_atomic_store((u64*)(seqp + 8), __builtin_amdgcn_s_memrealtime(), __ATOMIC_RELAXED, __HIP_MEMORY_SCOPE_AGENT);
    }
    STAMP(5);
  }
  if (dd != 0 && wv < 3) {
    char* stage = smem + LDS_STAGE + wv * 5120;
    const char* mbase = myws + WS_MSG + (size_t)(((call & 1) * 8 + src) * 16 + tile) * MSG_TILE;
    const __amdgpu_buffer_rsrc_t rm = __builtin_amdgcn_make_buffer_rsrc((void*)mbase, 0, MSG_TILE, 0x00020000);
    const __amdgpu_buffer_rsrc_t ro = __builtin_amdgcn_make_buffer_rsrc((void*)(P.out + (size_t)(src * M_SH + rho * BM) * N_SH), 0, BM * N_SH * 4, 0x00020000);
    for (int m = wv; m < 8; m += 3) {
      const int j = 2 * m + kap;
      const unsigned mo = (unsigned)(j * MSG_BYTES);
      unsigned val[16];
      unsigned n;
      for (;;) {
        u32x4 mm[NST];
#pragma unroll
        for (int s = 0; s < NST; ++s) {
          const unsigned o = s * 1024 + lane * 16;
          mm[s] = __builtin_amdgcn_raw_buffer_load_b128(rm, o < MSG_BYTES ? mo + o : OOB, 0, MLAUX);
        }
#pragma unroll
        for (int s = 0; s < NST; ++s) *(u32x4*)(stage + s * 1024 + lane * 16) = mm[s];
        asm volatile("s_waitcnt lgkmcnt(0)" ::: "memory");
        const unsigned mask = ((volatile unsigned short*)stage)[lane];
        const unsigned cnt = __builtin_popcount(mask);
        const unsigned incl = wave_scan(cnt);
        n = __builtin_amdgcn_readlane(incl, 63);
        unsigned ad = (unsigned)(size_t)(LDS_AS char*)(stage + 128) + (incl - cnt) * 4u;
        u64 bal[16];
        unsigned ms = mask << 16;
#pragma unroll
        for (int e = 0; e < 16; ++e) { asm volatile("v_add_co_u32 %0, %1, %0, %0" : "+v"(ms), "=s"(bal[e])); val[e] = 0x80000000u; }
#pragma unroll
        for (int q = 0; q < 4; ++q)
          asm volatile("s_mov_b64 exec, %5\n\tds_read_b32 %1, %0\n\tv_add_u32 %0, 4, %0\n\t"
                       "s_mov_b64 exec, %6\n\tds_read_b32 %2, %0\n\tv_add_u32 %0, 4, %0\n\t"
                       "s_mov_b64 exec, %7\n\tds_read_b32 %3, %0\n\tv_add_u32 %0, 4, %0\n\t"
                       "s_mov_b64 exec, %8\n\tds_read_b32 %4, %0\n\tv_add_u32 %0, 4, %0\n\t"
                       "s_mov_b64 exec, -1\n\ts_waitcnt lgkmcnt(0)"
                       : "+v"(ad), "+v"(val[4 * q]), "+v"(val[4 * q + 1]), "+v"(val[4 * q + 2]), "+v"(val[4 * q + 3])
                       : "s"(bal[4 * q]), "s"(bal[4 * q + 1]), "s"(bal[4 * q + 2]), "s"(bal[4 * q + 3]) : "memory");
        unsigned mn = (lane & 1) ? (mask & 1u) : 1u;
#pragma unroll
        for (int e = 0; e < 16; ++e) mn = min(mn, val[e]);
        if (__all(mn != 0u)) break;
        if (TIMED_OUT()) { bad = true; break; }
        __builtin_amdgcn_s_sleep(1);
      }
      const unsigned ovo = (unsigned)((j * 16 + 4 * g) * N_SH + strip * BN + 4 * c) * 4u;
#pragma unroll
      for (int rg = 0; rg < 4; ++rg) {
        u32x4 v;
#pragma unroll
        for (int t = 0; t < 4; ++t) v[t] = (unsigned)max((int)val[t * 4 + rg], 0);
        if (bad) v[0] = 0x7fc00000u;
        __builtin_amdgcn_raw_buffer_store_b128(v, ro, ovo + rg * (N_SH * 4), 0, OAUX);
      }
      const unsigned nb = (32u + n) * 4u;
#pragma unroll
      for (int s = 0; s < NST; ++s) {
        const unsigned o = s * 1024 + lane * 16;
        __builtin_amdgcn_raw_buffer_store_b128(u32x4{0u, 0u, 0u, 0u}, rm, o < nb ? mo + o : OOB, 0, 17);
      }
    }
  }
  STAMP(7);
  STAMP_FLUSH();
}

extern "C" void kernel_launch(int rank, int nranks, ncclComm_t comm, hipStream_t stream,
                              void* const* d_in, const int* in_sizes, int n_in,
                              void* d_out, int out_size, const int* in_dtypes, int out_dtype,
                              void* const* d_out_all, void* const* d_ws_all, size_t ws_bytes) {
  if (nranks != NRANK || (unsigned)rank >= (unsigned)NRANK || ws_bytes < WS_NEED) return;
  Params P;
  P.x = (const float*)d_in[0]; P.w = (const float*)d_in[1]; P.rank = rank; P.out = (float*)d_out;
  for (int p = 0; p < NRANK; ++p) P.ws[p] = (char*)d_ws_all[p];
  P.myws = P.ws[rank];
  HIP_CHECK(hipFuncSetAttribute((const void*)gemm_a2a, hipFuncAttributeMaxDynamicSharedMemorySize, LDS_BYTES));
  gemm_a2a<<<NWG, 256, LDS_BYTES, stream>>>(P);
  HIP_CHECK(hipGetLastError());
}
```

```cpp
#include <hip/hip_runtime.h>
#include <rccl/rccl.h>
#include <cstdio>

#define HIP_CHECK(expr) do { hipError_t _e = (expr); if (_e != hipSuccess) { \
    fprintf(stderr, "HIP error %s at %s:%d\n", hipGetErrorString(_e), __FILE__, __LINE__); \
    return; } } while (0)

typedef float f32x4 __attribute__((ext_vector_type(4)));
typedef _Float16 f16x2 __attribute__((ext_vector_type(2)));
typedef unsigned u32x4 __attribute__((ext_vector_type(4)));
typedef unsigned long long u64;
#define LDS_AS __attribute__((address_space(3)))

constexpr int NRANK = 8;
constexpr int M_SH = 512, K_DIM = 4096, N_DIM = 4096, N_SH = 512;
constexpr int BM = 256, BN = 64;
constexpr int NWG = 256;
#define EARLY 1
#define KC 22
#define XW KC
#define PACE 150
#define PACEWIN 300000ull
#define TSHIFT 300ull
#define MLIM 2560u
#define XCHAUX 16
#define OAUX 16
#define MLAUX 17
static_assert(KC == 22, "3 x 22 slots cover the 64 k-steps of a K half (the last slot of waves 1, 2 is empty)");
constexpr int NST = 5;
constexpr u64 DEADLINE_TICKS = 50000000ull;
constexpr int WS_XFLAG = 4096;
constexpr int WS_SEQ = 49152;
constexpr int WS_DBG = 65536;
constexpr int XGRP = 3 * 16 * KC * 1024;
constexpr int WS_X16 = 1 << 20;
constexpr int WS_XCH = 10 << 20;
constexpr int MSG_BYTES = 4224;
constexpr int WS_MSG = 18 << 20;
constexpr int MSG_TILE = 16 * MSG_BYTES;
constexpr size_t WS_NEED = (size_t)WS_MSG + 2ull * 8 * 16 * MSG_TILE;
constexpr int RED_BYTES = 3 * 4096;
constexpr int NRING = 8;
constexpr int LDS_PARK = NRING * RED_BYTES;
constexpr int LDS_STAGE = LDS_PARK + 8 * 5120;
constexpr int LDS_CNT = LDS_STAGE + 4 * 5120;
constexpr int LDS_BYTES = LDS_CNT + 256;
constexpr unsigned OOB = 0x7ffffff0u;

struct Params {
  const float* x; const float* w;
  float* out;
  char* myws;
  char* ws[NRANK];
  int rank;
};

__device__ __forceinline__ unsigned pk(float a, float b) {
  f16x2 r; r[0] = (_Float16)a; r[1] = (_Float16)b; return __builtin_bit_cast(unsigned, r);
}
__device__ __forceinline__ u32x4 make_srd(const void* p, unsigned bytes) {
  const u64 a = (u64)p;
  u32x4 r;
  r[0] = __builtin_amdgcn_readfirstlane((unsigned)a);
  r[1] = __builtin_amdgcn_readfirstlane((unsigned)(a >> 32)) & 0xffffu;
  r[2] = bytes; r[3] = 0x00020000u;
  return r;
}
__device__ __forceinline__ unsigned wave_scan(unsigned v) {
  v += __builtin_amdgcn_update_dpp(0u, v, 0x111, 0xf, 0xf, false);
  v += __builtin_amdgcn_update_dpp(0u, v, 0x112, 0xf, 0xf, false);
  v += __builtin_amdgcn_update_dpp(0u, v, 0x114, 0xf, 0xf, false);
  v += __builtin_amdgcn_update_dpp(0u, v, 0x118, 0xf, 0xf, false);
  v += __builtin_amdgcn_update_dpp(0u, v, 0x142, 0xa, 0xf, false);
  v += __builtin_amdgcn_update_dpp(0u, v, 0x143, 0xc, 0xf, false);
  return v;
}
#define STAMP(i)
#define STAMP_FLUSH()
#define STAMP_DECL
#define SB() __builtin_amdgcn_sched_barrier(0)
#define TIMED_OUT() (__builtin_amdgcn_s_memrealtime() - t_start > DEADLINE_TICKS)
#define SETBAD() do { bad = true; cnts[20] = 1u; } while (0)

__global__ void __launch_bounds__(256, 1) gemm_a2a(Params P) {
  extern __shared__ __attribute__((aligned(16))) char smem[];
  const int tid = threadIdx.x, lane = tid & 63, wv = __builtin_amdgcn_readfirstlane(tid >> 6);
  const int bid = blockIdx.x;
  const int idx = bid >> 3;
  const int kap = bid & 1, rho = idx & 1, dd = idx >> 2;
  const int strip = (bid & 6) + ((idx >> 1) & 1);
  const int dest = (P.rank + dd) & 7, src = (P.rank - dd) & 7;
  const int tile = rho * 8 + strip, pid = dd * 16 + tile;
  char* myws = P.myws;
  const int c = lane & 15, g = lane >> 4;
  STAMP_DECL
  STAMP(0);
  const u64 t_start = __builtin_amdgcn_s_memrealtime();
  volatile LDS_AS unsigned* cnts = (volatile LDS_AS unsigned*)(smem + LDS_CNT);
  const char* seqp = myws + WS_SEQ + 64 * bid;
  unsigned call;
  u32x4 seq;
  if (tid < 32) ((unsigned*)(smem + LDS_CNT))[tid] = 0u;

  const u32x4 rw = make_srd(P.w, K_DIM * N_DIM * 4);
  u32x4 rxs;
#define BLOAD(dst, vo, rs, so, imm, aux) asm volatile("buffer_load_dwordx4 %0, %1, %2, %3 offen offset:%4" aux : "=v"(dst) : "v"(vo), "s"(rs), "s"(so), "n"(imm) : "memory")
#define BSTORE(v, vo, rs, so, imm, aux) asm volatile("buffer_store_dwordx4 %0, %1, %2, %3 offen offset:%4" aux "\n\ts_nop 1" :: "v"(v), "v"(vo), "s"(rs), "s"(so), "n"(imm) : "memory")
#define WAIT1(n, a) asm volatile("s_waitcnt vmcnt(%1)" : "+v"(a) : "n"(n) : "memory")
#define WAIT2(n, a, b) asm volatile("s_waitcnt vmcnt(%2)" : "+v"(a), "+v"(b) : "n"(n) : "memory")
  const unsigned wvo = (unsigned)(4 * g * N_DIM + dest * N_SH + strip * BN + 4 * c) * 4u;
  const unsigned wso = (unsigned)(kap * 2048 + wv * 32) * (N_DIM * 4u);
  const unsigned lastoff = wv == 0 ? 0u : 0x40000000u;
  const unsigned wsoA = wso + rho * (16 * N_DIM * 4u), wsoB = wso + (1 - rho) * (16 * N_DIM * 4u);
  const unsigned xvo = lane * 16;
  bool bad = false;
  unsigned* const xflag = (unsigned*)(myws + WS_XFLAG + 8192 * (rho * 2 + kap) + 128 * lane);
#define WLOAD(i) BLOAD(wl[i], wvo, rw, (((i) & 4) ? wsoB : wsoA) + (unsigned)(((i) >> 3) * 96 + ((i) & 3)) * (N_DIM * 4u) + (((i) >> 3) == KC - 1 ? lastoff : 0u), 0, "")
#define XLOAD(s, q) BLOAD(win[s], xvo, rxs, (unsigned)(((q) >> 2) * 4096) + ((q) % KC == KC - 1 ? lastoff : 0u), ((q) & 3) * 1024, " sc1")
#define WCVT(i, jj) do { _Pragma("unroll") for (int t = 0; t < 4; ++t) { bb[t][jj] = pk(wl[i][t], wl[(i) + 1][t]); asm volatile("" : "+v"(bb[t][jj])); } SB(); } while (0)
#define WKEEP_(C, ks) do { _Pragma("unroll") for (int t = 0; t < 4; ++t) { u32x4 b_ = {bb[t][0], bb[t][1], bb[t][2], bb[t][3]}; \
    asm volatile("" : C(b_)); breg[ks][t] = b_; } SB(); } while (0)
#define WKEEPA(ks) WKEEP_("+a", ks)
#define WKEEPV(ks) WKEEP_("+v", ks)
#define SEQREQ() do { asm volatile("s_waitcnt lgkmcnt(0)" ::: "memory"); __builtin_amdgcn_s_barrier();                \
    asm volatile("s_load_dwordx4 %0, %1, 0x0" : "=s"(seq) : "s"(seqp) : "memory"); } while (0)
#define GETCALL() do { asm volatile("s_waitcnt lgkmcnt(0)" : "+s"(seq) :: "memory"); call = seq[0] + 1u;              \
    rxs = make_srd(myws + WS_X16 + (size_t)((call & 1) * 4 + rho * 2 + kap) * XGRP + wv * (16 * KC * 1024), 16 * KC * 1024u); \
    asm volatile("s_nop 4" ::: "memory"); } while (0)
#define XPOLL() do { while (!__all(__hip_atomic_load(xflag, __ATOMIC_RELAXED, __HIP_MEMORY_SCOPE_AGENT) >= call)) {   \
      if (TIMED_OUT()) { SETBAD(); break; }                                                                        \
      __builtin_amdgcn_s_sleep(4); } } while (0)
#define MF(i, CIN) "v_mfma_f32_16x16x32_f16 %[c" #i "], %[a], %[b" #i "], " CIN(i) "\n\t"
#define CIN_ACC(i) "%[c" #i "]"
#define CIN_ZERO(i) "0"
#define COUT(CC, A) [c0] CC(A[0]), [c1] CC(A[1]), [c2] CC(A[2]), [c3] CC(A[3])
#define BC_A "a"
#define BC_V "v"
#define BIN(K, ks) [b0] BC_##K(breg[ks][0]), [b1] BC_##K(breg[ks][1]), [b2] BC_##K(breg[ks][2]), [b3] BC_##K(breg[ks][3])
#define STEP_(K, CIN, CC, A, n_, s, sp, sn, so_, imm_, ks) do { asm volatile(MF(0, CIN)                               \
    "buffer_load_dwordx4 %[p], %[vo], %[rs], %[so] offen offset:%[imm] sc1\n\t" MF(1, CIN) MF(2, CIN)                \
    "s_waitcnt vmcnt(%[n])\n\t" MF(3, CIN)                                                                           \
    : COUT(CC, A), [p] "+v"(win[sp]), [nx] "+v"(win[sn])                                                             \
    : [a] "v"(win[s]), BIN(K, ks), [vo] "v"(xvo), [rs] "s"(rxs), [so] "s"(so_), [imm] "n"(imm_), [n] "n"(n_) : "memory"); SB(); } while (0)
#define MFMA_FENCE(A) asm volatile("s_nop 7\n\ts_nop 1" : "+v"(A[0]), "+v"(A[1]), "+v"(A[2]), "+v"(A[3]))

  const bool late = ((((unsigned)(size_t)P.w >> 8) + strip) & 1) == 0;
#define SHIFT_WAIT() do { if (late) { while (__builtin_amdgcn_s_memrealtime() - t_start < TSHIFT) __builtin_amdgcn_s_sleep(4); } } while (0)
  if (wv < 3) {
    f32x4 wl[8 * KC];
    u32x4 win[XW];
    u32x4 breg[KC][4];
    f32x4 eacc[EARLY][4];
    unsigned xc = 0;
#define P1_INIT()
#define P1_CLOAD() do { GETCALL(); asm volatile("global_load_dword %0, %1, off sc1" : "=v"(xc) : "v"(xflag) : "memory"); SB(); } while (0)
#define P1_CHECK() do { asm volatile("" : "+v"(xc)); if (!__all(xc >= call)) XPOLL(); asm volatile("" ::: "memory");  \
    SB(); STAMP(1); } while (0)
#define EMFMA_(K, CIN, CC, n_, s, e, ks) do { asm volatile("s_waitcnt vmcnt(%[n])\n\ts_nop 0\n\t" MF(0, CIN) MF(1, CIN) MF(2, CIN) MF(3, CIN) \
    : COUT(CC, eacc[e]), [a] "+v"(win[s]) : BIN(K, ks), [n] "n"(n_) : "memory"); SB(); } while (0)
#define EMFMA0(K, n_, s, e, ks) EMFMA_(K, CIN_ZERO, "=&v", n_, s, e, ks)
#define EMFMA(K, n_, s, e, ks) EMFMA_(K, CIN_ACC, "+v", n_, s, e, ks)
#define ESTEP(K, n_, s, sp, sn, q, e, ks) STEP_(K, CIN_ACC, "+v", eacc[e], n_, s, sp, sn, (unsigned)(((q) >> 2) * 4096) + ((q) % KC == KC - 1 ? lastoff : 0u), ((q) & 3) * 1024, ks)
    SEQREQ();
    SHIFT_WAIT();
    asm volatile("s_nop 4" ::: "memory");
    WLOAD(0);
    WLOAD(1);
    WLOAD(2);
    WLOAD(3);
    WLOAD(4);
    WLOAD(5);
    WLOAD(6);
    WLOAD(7);
    WLOAD(8);
    WLOAD(9);
    WLOAD(10);
    WLOAD(11);
    SB();
    P1_INIT();
    { unsigned bb[4][4];
    WAIT2(10, wl[0], wl[1]);
    WCVT(0, 0);
    WLOAD(12);
    WLOAD(13);
    SB();
    WAIT2(10, wl[2], wl[3]);
    WCVT(2, 1);
    WLOAD(14);
    WLOAD(15);
    SB();
    WAIT2(10, wl[4], wl[5]);
    WCVT(4, 2);
    WLOAD(16);
    WLOAD(17);
    SB();
    WAIT2(10, wl[6], wl[7]);
    WCVT(6, 3);
    WLOAD(18);
    WLOAD(19);
    SB();
    WKEEPA(0); }
    { unsigned bb[4][4];
    WAIT2(10, wl[8], wl[9]);
    WCVT(8, 0);
    WLOAD(20);
    WLOAD(21);
    SB();
    WAIT2(10, wl[10], wl[11]);
    WCVT(10, 1);
    WLOAD(22);
    WLOAD(23);
    SB();
    WAIT2(10, wl[12], wl[13]);
    WCVT(12, 2);
    WLOAD(24);
    WLOAD(25);
    SB();
    WAIT2(10, wl[14], wl[15]);
    WCVT(14, 3);
    WLOAD(26);
    WLOAD(27);
    SB();
    WKEEPA(1); }
    { unsigned bb[4][4];
    WAIT2(10, wl[16], wl[17]);
    WCVT(16, 0);
    WLOAD(28);
    WLOAD(29);
    SB();
    WAIT2(10, wl[18], wl[19]);
    WCVT(18, 1);
    WLOAD(30);
    WLOAD(31);
    SB();
    WAIT2(10, wl[20], wl[21]);
    WCVT(20, 2);
    WLOAD(32);
    WLOAD(33);
    SB();
    WAIT2(10, wl[22], wl[23]);
    WCVT(22, 3);
    WLOAD(34);
    WLOAD(35);
    SB();
    WKEEPA(2); }
    { unsigned bb[4][4];
    WAIT2(10, wl[24], wl[25]);
    WCVT(24, 0);
    WLOAD(36);
    WLOAD(37);
    SB();
    WAIT2(10, wl[26], wl[27]);
    WCVT(26, 1);
    WLOAD(38);
    WLOAD(39);
    SB();
    WAIT2(10, wl[28], wl[29]);
    WCVT(28, 2);
    WLOAD(40);
    WLOAD(41);
    SB();
    WAIT2(10, wl[30], wl[31]);
    WCVT(30, 3);
    WLOAD(42);
    WLOAD(43);
    SB();
    WKEEPA(3); }
    { unsigned bb[4][4];
    WAIT2(10, wl[32], wl[33]);
    WCVT(32, 0);
    WLOAD(44);
    WLOAD(45);
    SB();
    WAIT2(10, wl[34], wl[35]);
    WCVT(34, 1);
    WLOAD(46);
    WLOAD(47);
    SB();
    WAIT2(10, wl[36], wl[37]);
    WCVT(36, 2);
    WLOAD(48);
    WLOAD(49);
    SB();
    WAIT2(10, wl[38], wl[39]);
    WCVT(38, 3);
    WLOAD(50);
    WLOAD(51);
    SB();
    WKEEPA(4); }
    { unsigned bb[4][4];
    WAIT2(10, wl[40], wl[41]);
    WCVT(40, 0);
    WLOAD(52);
    WLOAD(53);
    SB();
    WAIT2(10, wl[42], wl[43]);
    WCVT(42, 1);
    WLOAD(54);
    WLOAD(55);
    SB();
    WAIT2(10, wl[44], wl[45]);
    WCVT(44, 2);
    WLOAD(56);
    WLOAD(57);
    SB();
    WAIT2(10, wl[46], wl[47]);
    WCVT(46, 3);
    WLOAD(58);
    WLOAD(59);
    SB();
    WKEEPA(5); }
    { unsigned bb[4][4];
    WAIT2(10, wl[48], wl[49]);
    WCVT(48, 0);
    WLOAD(60);
    WLOAD(61);
    SB();
    WAIT2(10, wl[50], wl[51]);
    WCVT(50, 1);
    WLOAD(62);
    WLOAD(63);
    SB();
    WAIT2(10, wl[52], wl[53]);
    WCVT(52, 2);
    WLOAD(64);
    WLOAD(65);
    SB();
    WAIT2(10, wl[54], wl[55]);
    WCVT(54, 3);
    WLOAD(66);
    WLOAD(67);
    SB();
    WKEEPA(6); }
    { unsigned bb[4][4];
    WAIT2(10, wl[56], wl[57]);
    WCVT(56, 0);
    WLOAD(68);
    WLOAD(69);
    SB();
    WAIT2(10, wl[58], wl[59]);
    WCVT(58, 1);
    WLOAD(70);
    WLOAD(71);
    SB();
    WAIT2(10, wl[60], wl[61]);
    WCVT(60, 2);
    WLOAD(72);
    WLOAD(73);
    SB();
    WAIT2(10, wl[62], wl[63]);
    WCVT(62, 3);
    WLOAD(74);
    WLOAD(75);
    SB();
    WKEEPA(7); }
    { unsigned bb[4][4];
    WAIT2(10, wl[64], wl[65]);
    WCVT(64, 0);
    WLOAD(76);
    WLOAD(77);
    SB();
    WAIT2(10, wl[66], wl[67]);
    WCVT(66, 1);
    WLOAD(78);
    WLOAD(79);
    SB();
    WAIT2(10, wl[68], wl[69]);
    WCVT(68, 2);
    WLOAD(80);
    WLOAD(81);
    SB();
    WAIT2(10, wl[70], wl[71]);
    WCVT(70, 3);
    WLOAD(82);
    WLOAD(83);
    SB();
    WKEEPA(8); }
    { unsigned bb[4][4];
    WAIT2(10, wl[72], wl[73]);
    WCVT(72, 0);
    WLOAD(84);
    WLOAD(85);
    SB();
    WAIT2(10, wl[74], wl[75]);
    WCVT(74, 1);
    WLOAD(86);
    WLOAD(87);
    SB();
    WAIT2(10, wl[76], wl[77]);
    WCVT(76, 2);
    WLOAD(88);
    WLOAD(89);
    SB();
    WAIT2(10, wl[78], wl[79]);
    WCVT(78, 3);
    WLOAD(90);
    WLOAD(91);
    SB();
    WKEEPA(9); }
    { unsigned bb[4][4];
    WAIT2(10, wl[80], wl[81]);
    WCVT(80, 0);
    WLOAD(92);
    WLOAD(93);
    SB();
    WAIT2(10, wl[82], wl[83]);
    WCVT(82, 1);
    WLOAD(94);
    WLOAD(95);
    SB();
    WAIT2(10, wl[84], wl[85]);
    WCVT(84, 2);
    WLOAD(96);
    WLOAD(97);
    SB();
    WAIT2(10, wl[86], wl[87]);
    WCVT(86, 3);
    WLOAD(98);
    WLOAD(99);
    SB();
    WKEEPA(10); }
    { unsigned bb[4][4];
    WAIT2(10, wl[88], wl[89]);
    WCVT(88, 0);
    WLOAD(100);
    WLOAD(101);
    SB();
    WAIT2(10, wl[90], wl[91]);
    WCVT(90, 1);
    WLOAD(102);
    WLOAD(103);
    SB();
    WAIT2(10, wl[92], wl[93]);
    WCVT(92, 2);
    WLOAD(104);
    WLOAD(105);
    SB();
    WAIT2(10, wl[94], wl[95]);
    WCVT(94, 3);
    WLOAD(106);
    WLOAD(107);
    SB();
    WKEEPA(11); }
    { unsigned bb[4][4];
    WAIT2(10, wl[96], wl[97]);
    WCVT(96, 0);
    WLOAD(108);
    WLOAD(109);
    SB();
    WAIT2(10, wl[98], wl[99]);
    WCVT(98, 1);
    WLOAD(110);
    WLOAD(111);
    SB();
    WAIT2(10, wl[100], wl[101]);
    WCVT(100, 2);
    WLOAD(112);
    WLOAD(113);
    SB();
    WAIT2(10, wl[102], wl[103]);
    WCVT(102, 3);
    WLOAD(114);
    WLOAD(115);
    SB();
    WKEEPA(12); }
    { unsigned bb[4][4];
    WAIT2(10, wl[104], wl[105]);
    WCVT(104, 0);
    WLOAD(116);
    WLOAD(117);
    SB();
    WAIT2(10, wl[106], wl[107]);
    WCVT(106, 1);
    WLOAD(118);
    WLOAD(119);
    SB();
    WAIT2(10, wl[108], wl[109]);
    WCVT(108, 2);
    WLOAD(120);
    WLOAD(121);
    SB();
    WAIT2(10, wl[110], wl[111]);
    WCVT(110, 3);
    WLOAD(122);
    WLOAD(123);
    SB();
    WKEEPA(13); }
    { unsigned bb[4][4];
    WAIT2(10, wl[112], wl[113]);
    WCVT(112, 0);
    WLOAD(124);
    WLOAD(125);
    SB();
    WAIT2(10, wl[114], wl[115]);
    WCVT(114, 1);
    WLOAD(126);
    WLOAD(127);
    SB();
    WAIT2(10, wl[116], wl[117]);
    WCVT(116, 2);
    WLOAD(128);
    WLOAD(129);
    SB();
    WAIT2(10, wl[118], wl[119]);
    WCVT(118, 3);
    WLOAD(130);
    WLOAD(131);
    SB();
    WKEEPA(14); }
    { unsigned bb[4][4];
    WAIT2(10, wl[120], wl[121]);
    WCVT(120, 0);
    WLOAD(132);
    WLOAD(133);
    SB();
    WAIT2(10, wl[122], wl[123]);
    WCVT(122, 1);
    WLOAD(134);
    WLOAD(135);
    SB();
    WAIT2(10, wl[124], wl[125]);
    WCVT(124, 2);
    WLOAD(136);
    WLOAD(137);
    SB();
    WAIT2(10, wl[126], wl[127]);
    WCVT(126, 3);
    WLOAD(138);
    WLOAD(139);
    SB();
    WKEEPA(15); }
    { unsigned bb[4][4];
    WAIT2(10, wl[128], wl[129]);
    WCVT(128, 0);
    WLOAD(140);
    WLOAD(141);
    SB();
    WAIT2(10, wl[130], wl[131]);
    WCVT(130, 1);
    WLOAD(142);
    WLOAD(143);
    SB();
    WAIT2(10, wl[132], wl[133]);
    WCVT(132, 2);
    WLOAD(144);
    WLOAD(145);
    SB();
    WAIT2(10, wl[134], wl[135]);
    WCVT(134, 3);
    WLOAD(146);
    WLOAD(147);
    SB();
    WKEEPV(16); }
    { unsigned bb[4][4];
    WAIT2(10, wl[136], wl[137]);
    WCVT(136, 0);
    WLOAD(148);
    WLOAD(149);
    SB();
    P1_CLOAD();
    WAIT2(11, wl[138], wl[139]);
    WCVT(138, 1);
    WLOAD(150);
    WLOAD(151);
    SB();
    WAIT2(11, wl[140], wl[141]);
    WCVT(140, 2);
    WLOAD(152);
    WLOAD(153);
    SB();
    WAIT2(11, wl[142], wl[143]);
    WCVT(142, 3);
    WLOAD(154);
    WLOAD(155);
    SB();
    WKEEPV(17); }
    { unsigned bb[4][4];
    WAIT2(11, wl[144], wl[145]);
    WCVT(144, 0);
    WLOAD(156);
    WLOAD(157);
    SB();
    WAIT2(11, wl[146], wl[147]);
    WCVT(146, 1);
    WLOAD(158);
    WLOAD(159);
    SB();
    WAIT2(11, wl[148], wl[149]);
    WCVT(148, 2);
    WLOAD(160);
    WLOAD(161);
    SB();
    WAIT2(10, wl[150], wl[151]);
    WCVT(150, 3);
    WLOAD(162);
    WLOAD(163);
    SB();
    WKEEPV(18); }
    P1_CHECK();
    XLOAD(0, 0);
    XLOAD(1, 1);
    { unsigned bb[4][4];
    WAIT2(12, wl[152], wl[153]);
    WCVT(152, 0);
    WLOAD(164);
    WLOAD(165);
    SB();
    XLOAD(2, 2);
    XLOAD(3, 3);
    WAIT2(14, wl[154], wl[155]);
    WCVT(154, 1);
    WLOAD(166);
    WLOAD(167);
    SB();
    XLOAD(4, 4);
    XLOAD(5, 5);
    WAIT2(16, wl[156], wl[157]);
    WCVT(156, 2);
    WLOAD(168);
    WLOAD(169);
    SB();
    XLOAD(6, 6);
    XLOAD(7, 7);
    WAIT2(18, wl[158], wl[159]);
    WCVT(158, 3);
    WLOAD(170);
    WLOAD(171);
    SB();
    WKEEPV(19); }
    XLOAD(8, 8);
    XLOAD(9, 9);
    { unsigned bb[4][4];
    WAIT2(20, wl[160], wl[161]);
    WCVT(160, 0);
    WLOAD(172);
    WLOAD(173);
    SB();
    XLOAD(10, 10);
    XLOAD(11, 11);
    WAIT2(22, wl[162], wl[163]);
    WCVT(162, 1);
    WLOAD(174);
    WLOAD(175);
    SB();
    EMFMA0(A, 23, 0, 0, 0);
    EMFMA(A, 22, 1, 0, 1);
    XLOAD(12, 12);
    XLOAD(13, 13);
    WAIT2(22, wl[164], wl[165]);
    WCVT(164, 2);
    SB();
    EMFMA(A, 21, 2, 0, 2);
    EMFMA(A, 20, 3, 0, 3);
    XLOAD(14, 14);
    XLOAD(15, 15);
    WAIT2(20, wl[166], wl[167]);
    WCVT(166, 3);
    SB();
    WKEEPV(20); }
    EMFMA(A, 19, 4, 0, 4);
    EMFMA(A, 18, 5, 0, 5);
    XLOAD(16, 16);
    XLOAD(17, 17);
    { unsigned bb[4][4];
    WAIT2(18, wl[168], wl[169]);
    WCVT(168, 0);
    SB();
    EMFMA(A, 17, 6, 0, 6);
    EMFMA(A, 16, 7, 0, 7);
    XLOAD(18, 18);
    XLOAD(19, 19);
    WAIT2(16, wl[170], wl[171]);
    WCVT(170, 1);
    SB();
    EMFMA(A, 15, 8, 0, 8);
    EMFMA(A, 14, 9, 0, 9);
    XLOAD(20, 20);
    XLOAD(21, 21);
    WAIT2(14, wl[172], wl[173]);
    WCVT(172, 2);
    SB();
    EMFMA(A, 13, 10, 0, 10);
    EMFMA(A, 12, 11, 0, 11);
    XLOAD(0, 22);
    XLOAD(1, 23);
    WAIT2(12, wl[174], wl[175]);
    WCVT(174, 3);
    SB();
    WKEEPV(21); }
    EMFMA(A, 11, 12, 0, 12);
    EMFMA(A, 10, 13, 0, 13);
    XLOAD(2, 24);
    XLOAD(3, 25);
    STAMP(3);
    EMFMA(A, 11, 14, 0, 14);
    XLOAD(4, 26);
    XLOAD(5, 27);
    EMFMA(A, 12, 15, 0, 15);
    XLOAD(6, 28);
    XLOAD(7, 29);
    EMFMA(V, 13, 16, 0, 16);
    XLOAD(8, 30);
    XLOAD(9, 31);
    EMFMA(V, 14, 17, 0, 17);
    XLOAD(10, 32);
    XLOAD(11, 33);
    EMFMA(V, 15, 18, 0, 18);
    XLOAD(12, 34);
    XLOAD(13, 35);
    EMFMA(V, 16, 19, 0, 19);
    XLOAD(14, 36);
    XLOAD(15, 37);
    EMFMA(V, 17, 20, 0, 20);
    XLOAD(16, 38);
    XLOAD(17, 39);
    XLOAD(18, 40);
    XLOAD(19, 41);
    WAIT1(20, win[21]);
    asm volatile("s_nop 1" ::: "memory");
    ESTEP(V, 20, 21, 20, 0, 42, 0, 21);
    STAMP(2);
    static_assert(EARLY == 1, "chunk 0 comes out of phase 1");
    const bool paced = t_start - (((u64)seq[3] << 32) | seq[2]) < PACEWIN;
    u64 tchunk = __builtin_amdgcn_s_memrealtime();
    const unsigned cnt_ad = (unsigned)(size_t)(LDS_AS char*)(smem + LDS_CNT);
    f32x4 accB[4];
#define PUT_CHECK(o_) do { if ((int)__builtin_amdgcn_readfirstlane(consumed) < (o_) - (NRING - 1)) {     \
        while ((int)cnts[16] < (o_) - (NRING - 1)) { if (TIMED_OUT()) { SETBAD(); break; } } }                      \
      asm volatile("" ::: "memory"); SB(); } while (0)
#define PUT_SIGNAL(o_) do { asm volatile("s_mov_b64 exec, 1\n\tds_add_u32 %0, %1\n\ts_mov_b64 exec, -1" :: "v"(cnt_ad + (unsigned)(o_) * 4u), "v"(1u) : "memory"); SB(); } while (0)
#define KSTEPS(AC, K0, K1)                                                                                           \
      _Pragma("unroll") for (int k = K0; k < K1; ++k) {                                                              \
        if (k == 0) STEP_(A, CIN_ZERO, "=&v", AC, XW - 2, k % XW, (k + XW - 1) % XW, (k + 1) % XW, qb + (k >> 2) * 4096 + ((k + XW - 1) % KC == KC - 1 ? lastoff : 0u), (k & 3) * 1024, k); \
        else if (k < 16) STEP_(A, CIN_ACC, "+v", AC, XW - 2, k % XW, (k + XW - 1) % XW, (k + 1) % XW, qb + (k >> 2) * 4096 + ((k + XW - 1) % KC == KC - 1 ? lastoff : 0u), (k & 3) * 1024, k); \
        else STEP_(V, CIN_ACC, "+v", AC, XW - 2, k % XW, (k + XW - 1) % XW, (k + 1) % XW, qb + (k >> 2) * 4096 + ((k + XW - 1) % KC == KC - 1 ? lastoff : 0u), (k & 3) * 1024, k); \
        if (k >= 2 && k < 6) { pw[(k - 2) * 64] = PVX[k - 2]; SB(); }                                                \
        if (k == 6) PUT_SIGNAL(o - 1);                                                                               \
        if (k == 12) { consumed = cnts[16]; SB(); }                                                                  \
      }
#define CHUNK(AC, PV, o_) do { const int o = (o_);                                                                   \
      const unsigned qb = (unsigned)(o * KC + XW - 1) * 1024u;           \
      f32x4* pw = (f32x4*)(smem + ((o - 1) % NRING) * RED_BYTES + wv * 4096 + xvo);                                  \
      f32x4 (&PVX)[4] = PV;                                                                                          \
      KSTEPS(AC, 0, 2)                                                                                               \
      PUT_CHECK(o - 1);                                                                                              \
      KSTEPS(AC, 2, KC)                                                                                              \
      if (paced) { tchunk += PACE; while (__builtin_amdgcn_s_memrealtime() < tchunk) __builtin_amdgcn_s_sleep(1); }  \
    } while (0)
    unsigned consumed = 0;
    for (int i = 0; i < 7; ++i) {
      CHUNK(accB, eacc[0], 2 * i + 1);
      CHUNK(eacc[0], accB, 2 * i + 2);
    }
    CHUNK(accB, eacc[0], 15);
    MFMA_FENCE(accB);
    PUT_CHECK(15);
    {
      f32x4* pw = (f32x4*)(smem + (15 % NRING) * RED_BYTES + wv * 4096 + xvo);
#pragma unroll
      for (int t = 0; t < 4; ++t) pw[t * 64] = accB[t];
      asm volatile("" ::: "memory");
      PUT_SIGNAL(15);
    }
    STAMP(4);
    asm volatile("s_waitcnt vmcnt(0)" ::: "memory");
#pragma unroll
    for (int i = 0; i < XW; ++i) asm volatile("" :: "v"(win[i]));
  } else {
    SEQREQ();
    GETCALL();
    for (int cb = 0; cb < (late ? 2 : 0); ++cb) {
      const int blk = bid ^ (16 * cb), rc = blk >> 3, kb = blk & 7;
      const u32x4 rx = make_srd(P.x, M_SH * K_DIM * 4);
      const unsigned xcvo = (unsigned)((rc * 16 + c) * K_DIM + kb * 512 + g * 4) * 4u;
      const unsigned xcvA = xcvo + (rc >> 4) * 64u, xcvB = xcvo + 64u - (rc >> 4) * 64u;
      f32x4 xl[32];
      asm volatile("s_nop 4" ::: "memory");
#pragma unroll
      for (int i = 0; i < 32; ++i) BLOAD(xl[i], (i & 1) ? xcvB : xcvA, rx, 0u, (i >> 1) * 128, "");
      const int grp = (rc >> 4) * 2 + (kb >> 2), oc = (rc & 15) ^ ((kb >> 2) ^ 1);
      const u32x4 rx16w = make_srd(myws + WS_X16 + (size_t)((call & 1) * 4 + grp) * XGRP, XGRP);
      asm volatile("s_nop 4" ::: "memory");
#pragma unroll
      for (int i = 0; i < 16; ++i) {
        const int gk = (kb & 3) * 16 + i;
        const int pos = ((gk % 3) * 16 + oc) * KC + gk / 3;
        WAIT2(30 - i, xl[2 * i], xl[2 * i + 1]);
        const f32x4 a_ = xl[2 * i], b_ = xl[2 * i + 1];
        u32x4 o_ = {pk(a_[0], a_[1]), pk(a_[2], a_[3]), pk(b_[0], b_[1]), pk(b_[2], b_[3])};
        BSTORE(o_, xvo, rx16w, (unsigned)pos * 1024u, 0, " sc1");
      }
      asm volatile("s_waitcnt vmcnt(0)" ::: "memory");
      if (lane == 0) {
        unsigned* f_ = (unsigned*)(myws + WS_XFLAG + 8192 * grp + 128 * ((rc & 15) * 4 + (kb & 3)));
        asm volatile("global_store_dword %0, %1, off sc1" :: "v"(f_), "v"(call) : "memory");
      }
      SB();
    }
    STAMP(6);
  }
  {
    const bool local = dd == 0;
    char* dstp = local ? (char*)(P.out + (size_t)(P.rank * M_SH + rho * BM) * N_SH)
                       : P.ws[dest] + WS_MSG + (size_t)(((call & 1) * 8 + P.rank) * 16 + tile) * MSG_TILE;
    const __amdgpu_buffer_rsrc_t rxch = __builtin_amdgcn_make_buffer_rsrc((void*)(myws + WS_XCH + (size_t)pid * 65536), 0, 65536, 0x00020000);
    char* const buf = smem + LDS_STAGE + wv * 5120;
    const unsigned tag = call & 1u;

    auto gather = [&](int o, f32x4 (&acc)[4]) {
      while (cnts[o] < 3u) { if (TIMED_OUT()) { SETBAD(); break; } __builtin_amdgcn_s_sleep(1); }
      asm volatile("" ::: "memory");
      const f32x4* part = (const f32x4*)(smem + (o % NRING) * RED_BYTES + xvo);
      f32x4 pr[12];
#pragma unroll
      for (int i = 0; i < 12; ++i) pr[i] = part[i * 64];
      SB();
#pragma unroll
      for (int t = 0; t < 4; ++t) {
        acc[t] = pr[t] + (pr[4 + t] + pr[8 + t]);
        asm volatile("" : "+v"(acc[t]) :: "memory");
      }
      if (cnts[20] != 0u) acc[0][0] = __builtin_nanf("");
      if (lane == 0 && o < 15) cnts[16] = (unsigned)(o + 1);
      SB();
    };
    auto finalize = [&](int j, f32x4 (&a)[4], u32x4 (&pp)[4]) {
      const unsigned xo = (unsigned)j * 4096u;
      auto complete = [&]() {
        unsigned an = ~0u, orr = 0u;
#pragma unroll
        for (int t = 0; t < 4; ++t)
#pragma unroll
          for (int rg = 0; rg < 4; ++rg) { an &= pp[t][rg]; orr |= pp[t][rg]; }
        return __all(((tag ? an : ~orr) & 1u) != 0u);
      };
      if (!complete()) {
        do {
          if (TIMED_OUT()) { SETBAD(); break; }
#pragma unroll
          for (int t = 0; t < 4; ++t) pp[t] = __builtin_amdgcn_raw_buffer_load_b128(rxch, xvo + t * 1024, xo, XCHAUX);
          asm volatile("" ::: "memory");
        } while (!complete());
      }
#pragma unroll
      for (int t = 0; t < 4; ++t) a[t] += __builtin_bit_cast(f32x4, pp[t] & ~1u);
      if (bad) a[0][0] = __builtin_nanf("");
      if (local) {
        const __amdgpu_buffer_rsrc_t rd = __builtin_amdgcn_make_buffer_rsrc((void*)dstp, 0, BM * N_SH * 4, 0x00020000);
        const unsigned ovo = (xvo >> 8) * (4 * N_SH * 4) + (xvo & 255u);
        const unsigned so = (unsigned)(j * 16 * N_SH + strip * BN) * 4u;
#pragma unroll
        for (int rg = 0; rg < 4; ++rg) {
          f32x4 v;
#pragma unroll
          for (int t = 0; t < 4; ++t) v[t] = a[t][rg] <= 0.f ? 0.f : a[t][rg];
          __builtin_amdgcn_raw_buffer_store_b128(__builtin_bit_cast(u32x4, v), rd, ovo + so + rg * (N_SH * 4), 0, OAUX);
        }
      } else {
        u64 bal[16];
        unsigned bits[16];
#pragma unroll
        for (int e = 0; e < 16; ++e) {
          const float av = a[e >> 2][e & 3];
          bal[e] = __builtin_amdgcn_ballot_w64(!(av <= 0.f));
          bits[e] = __builtin_bit_cast(unsigned, av) & 0x7fffffffu;
        }
        asm volatile("s_nop 1\n\tv_cndmask_b32 %0, %1, %0, %2" : "+v"(bits[15]) : "v"(0x80000000u), "s"(bal[15]));
        bal[15] |= 0xaaaaaaaaaaaaaaaaull;
        unsigned mask = 0;
        asm volatile("s_nop 1" ::: "memory");
#pragma unroll
        for (int e = 0; e < 16; ++e)
          asm volatile("v_addc_co_u32 %0, vcc, %0, %0, %1" : "+v"(mask) : "s"(bal[e]) : "vcc");
        const unsigned cnt = __builtin_popcount(mask);
        const unsigned incl = wave_scan(cnt);
        const unsigned n = __builtin_amdgcn_readlane(incl, 63);
        unsigned ad = (unsigned)(size_t)(LDS_AS char*)(buf + 128) + (incl - cnt) * 4u;
#pragma unroll
        for (int q = 0; q < 4; ++q)
          asm volatile("s_mov_b64 exec, %5\n\tds_write_b32 %0, %1\n\tv_add_u32 %0, 4, %0\n\t"
                       "s_mov_b64 exec, %6\n\tds_write_b32 %0, %2\n\tv_add_u32 %0, 4, %0\n\t"
                       "s_mov_b64 exec, %7\n\tds_write_b32 %0, %3\n\tv_add_u32 %0, 4, %0\n\t"
                       "s_mov_b64 exec, %8\n\tds_write_b32 %0, %4\n\tv_add_u32 %0, 4, %0\n\t"
                       "s_mov_b64 exec, -1"
                       : "+v"(ad) : "v"(bits[4 * q]), "v"(bits[4 * q + 1]), "v"(bits[4 * q + 2]), "v"(bits[4 * q + 3]),
                         "s"(bal[4 * q]), "s"(bal[4 * q + 1]), "s"(bal[4 * q + 2]), "s"(bal[4 * q + 3]) : "memory");
        *(unsigned short*)(buf + (xvo >> 3)) = (unsigned short)mask;
        const __amdgpu_buffer_rsrc_t rd = __builtin_amdgcn_make_buffer_rsrc((void*)(dstp + j * MSG_BYTES), 0, (int)((32u + n) * 4u), 0x00020000);
        asm volatile("s_waitcnt lgkmcnt(0)" ::: "memory");
        u32x4 v[NST];
#pragma unroll
        for (int s = 0; s < NST; ++s) v[s] = *(u32x4*)(buf + s * 1024 + xvo);
#pragma unroll
        for (int s = 0; s < NST; ++s) __builtin_amdgcn_raw_buffer_store_b128(v[s], rd, xvo + s * 1024, 0, 17);
      }
      SB();
    };

    f32x4 held[4];
#pragma unroll
    for (int t = 0; t < 4; ++t) held[t] = f32x4{0.f, 0.f, 0.f, 0.f};
    if (wv == 3) {
      for (int it = 0; it < 8; ++it) {
        const int jA = 2 * it + (kap ^ 1), jP = 2 * it - 2 + kap;
        u32x4 pp[4];
        const unsigned po = it > 0 ? (unsigned)jP * 4096u : OOB - 4096u;
#pragma unroll
        for (int t = 0; t < 4; ++t) pp[t] = __builtin_amdgcn_raw_buffer_load_b128(rxch, xvo + t * 1024, po, XCHAUX);
        asm volatile("" ::: "memory");
        f32x4 acc[4];
        gather(2 * it, acc);
#pragma unroll
        for (int t = 0; t < 4; ++t) {
          u32x4 v = __builtin_bit_cast(u32x4, acc[t]);
#pragma unroll
          for (int rg = 0; rg < 4; ++rg) v[rg] = (v[rg] & ~1u) | tag;
          __builtin_amdgcn_raw_buffer_store_b128(v, rxch, xvo + (unsigned)jA * 4096u + t * 1024, 0, 16);
        }
        SB();
        asm volatile("" : "+v"(pp[0]), "+v"(pp[1]), "+v"(pp[2]), "+v"(pp[3]));
        if (it > 0) finalize(jP, held, pp);
        if (it == 1) STAMP(3);
        if (it < 7) gather(2 * it + 1, held);
      }
      STAMP(4);
      if (lane == 0) {
        __hip_atomic_store((unsigned*)seqp, call, __ATOMIC_RELAXED, __HIP_MEMORY_SCOPE_AGENT);
        __hip_atomic_store((u64*)(seqp + 8), __builtin_amdgcn_s_memrealtime(), __ATOMIC_RELAXED, __HIP_MEMORY_SCOPE_AGENT);
      }
    } else if (wv == 2) {
      u32x4 pp[4];
#pragma unroll
      for (int t = 0; t < 4; ++t) pp[t] = __builtin_amdgcn_raw_buffer_load_b128(rxch, xvo + t * 1024, (unsigned)(14 + kap) * 4096u, XCHAUX);
      asm volatile("" ::: "memory");
      gather(15, held);
      finalize(14 + kap, held, pp);
    }
    STAMP(5);
  }
  if (dd != 0) {
    char* stage = smem + LDS_STAGE + wv * 5120;
    const char* mbase = myws + WS_MSG + (size_t)(((call & 1) * 8 + src) * 16 + tile) * MSG_TILE;
    const __amdgpu_buffer_rsrc_t rm = __builtin_amdgcn_make_buffer_rsrc((void*)mbase, 0, MSG_TILE, 0x00020000);
    const __amdgpu_buffer_rsrc_t ro = __builtin_amdgcn_make_buffer_rsrc((void*)(P.out + (size_t)(src * M_SH + rho * BM) * N_SH), 0, BM * N_SH * 4, 0x00020000);
    unsigned mlim = MLIM;
    auto mload = [&](int j, u32x4 (&mm)[NST]) {
#pragma unroll
      for (int s = 0; s < NST; ++s) {
        const unsigned o = s * 1024 + lane * 16;
        mm[s] = __builtin_amdgcn_raw_buffer_load_b128(rm, o < mlim ? (unsigned)(j * MSG_BYTES) + o : OOB, 0, MLAUX);
      }
      asm volatile("" ::: "memory");
    };
    auto park = [&](char* st, u32x4 (&mm)[NST]) {
#pragma unroll
      for (int s = 0; s < NST; ++s) *(u32x4*)(st + s * 1024 + lane * 16) = mm[s];
    };
    auto decode = [&](char* stage, unsigned (&val)[16], unsigned& n) {
      asm volatile("s_waitcnt lgkmcnt(0)" ::: "memory");
      const unsigned mask = ((volatile LDS_AS unsigned short*)stage)[lane];
      const unsigned cnt = __builtin_popcount(mask);
      const unsigned incl = wave_scan(cnt);
      n = __builtin_amdgcn_readlane(incl, 63);
      unsigned ad = (unsigned)(size_t)(LDS_AS char*)(stage + 128) + (incl - cnt) * 4u;
      u64 bal[16];
      unsigned ms = mask << 16;
#pragma unroll
      for (int e = 0; e < 16; ++e) { asm volatile("v_add_co_u32 %0, %1, %0, %0" : "+v"(ms), "=s"(bal[e])); val[e] = 0x80000000u; }
#pragma unroll
      for (int q = 0; q < 4; ++q)
        asm volatile("s_mov_b64 exec, %5\n\tds_read_b32 %1, %0\n\tv_add_u32 %0, 4, %0\n\t"
                     "s_mov_b64 exec, %6\n\tds_read_b32 %2, %0\n\tv_add_u32 %0, 4, %0\n\t"
                     "s_mov_b64 exec, %7\n\tds_read_b32 %3, %0\n\tv_add_u32 %0, 4, %0\n\t"
                     "s_mov_b64 exec, %8\n\tds_read_b32 %4, %0\n\tv_add_u32 %0, 4, %0\n\t"
                     "s_mov_b64 exec, -1"
                     : "+v"(ad), "+v"(val[4 * q]), "+v"(val[4 * q + 1]), "+v"(val[4 * q + 2]), "+v"(val[4 * q + 3])
                     : "s"(bal[4 * q]), "s"(bal[4 * q + 1]), "s"(bal[4 * q + 2]), "s"(bal[4 * q + 3]) : "memory");
      asm volatile("s_waitcnt lgkmcnt(0)"
                   : "+v"(val[0]), "+v"(val[1]), "+v"(val[2]), "+v"(val[3]), "+v"(val[4]), "+v"(val[5]), "+v"(val[6]), "+v"(val[7]),
                     "+v"(val[8]), "+v"(val[9]), "+v"(val[10]), "+v"(val[11]), "+v"(val[12]), "+v"(val[13]), "+v"(val[14]), "+v"(val[15]) :: "memory");
      unsigned mn = (lane & 1) ? (mask & 1u) : 1u;
#pragma unroll
      for (int e = 0; e < 16; ++e) mn = min(mn, val[e]);
      return (bool)__all(mn != 0u);
    };
    auto emit = [&](int j, const unsigned (&val)[16], unsigned n) {
      const unsigned ovo = (unsigned)((j * 16 + 4 * g) * N_SH + strip * BN + 4 * c) * 4u;
#pragma unroll
      for (int rg = 0; rg < 4; ++rg) {
        u32x4 v;
#pragma unroll
        for (int t = 0; t < 4; ++t) v[t] = (unsigned)max((int)val[t * 4 + rg], 0);
        if (bad) v[0] = 0x7fc00000u;
        __builtin_amdgcn_raw_buffer_store_b128(v, ro, ovo + rg * (N_SH * 4), 0, OAUX);
      }
      const unsigned nb = (32u + n) * 4u;
#pragma unroll
      for (int s = 0; s < NST; ++s) {
        const unsigned o = s * 1024 + lane * 16;
        __builtin_amdgcn_raw_buffer_store_b128(u32x4{0u, 0u, 0u, 0u}, rm, o < nb ? (unsigned)(j * MSG_BYTES) + o : OOB, 0, 17);
      }
      asm volatile("" ::: "memory");
    };
    unsigned pend = wv < 2 ? 0x45u << wv : 4u << wv;
    unsigned parked = 0;
    if (wv < 2) {
      u32x4 mm[3][NST];
#pragma unroll
      for (int h = 0; h < 3; ++h) mload(2 * (wv + (h == 2 ? 6 : 2 * h)) + kap, mm[h]);
#pragma unroll
      for (int h = 0; h < 3; ++h) park(smem + LDS_PARK + (wv + (h == 2 ? 6 : 2 * h)) * 5120, mm[h]);
      parked = pend;
    }
    while (pend) {
      const int m = __builtin_ctz(pend);
      char* st = wv < 2 ? smem + LDS_PARK + m * 5120 : stage;
      if (!((parked >> m) & 1u)) {
        u32x4 mm[NST];
        mload(2 * m + kap, mm);
        park(st, mm);
      }
      parked &= ~(1u << m);
      unsigned val[16], n;
      bool ok = decode(st, val, n);
      if (!ok && TIMED_OUT()) { bad = true; ok = true; }
      if (ok) { emit(2 * m + kap, val, n); pend &= pend - 1u; }
      else { if ((32u + n) * 4u > mlim) mlim = MSG_BYTES; __builtin_amdgcn_s_sleep(1); }
    }
  }
  STAMP(7);
  STAMP_FLUSH();
}

extern "C" void kernel_launch(int rank, int nranks, ncclComm_t comm, hipStream_t stream,
                              void* const* d_in, const int* in_sizes, int n_in,
                              void* d_out, int out_size, const int* in_dtypes, int out_dtype,
                              void* const* d_out_all, void* const* d_ws_all, size_t ws_bytes) {
  if (nranks != NRANK || (unsigned)rank >= (unsigned)NRANK || ws_bytes < WS_NEED) return;
  Params P;
  P.x = (const float*)d_in[0]; P.w = (const float*)d_in[1]; P.rank = rank; P.out = (float*)d_out;
  for (int p = 0; p < NRANK; ++p) P.ws[p] = (char*)d_ws_all[p];
  P.myws = P.ws[rank];
  HIP_CHECK(hipFuncSetAttribute((const void*)gemm_a2a, hipFuncAttributeMaxDynamicSharedMemorySize, LDS_BYTES));
  gemm_a2a<<<NWG, 256, LDS_BYTES, stream>>>(P);
  HIP_CHECK(hipGetLastError());
}
```
